```python
import numpy as np

kernel_name = "gpu_gemm_a2a_m4096_k4096_n16384_bf16_none"


ABSMAX_RTOL = 0.008

COMPUTE_DTYPE = "bfloat16"
EPILOGUE = "none"
IN_QUANT = None
OUT_QUANT = None
IN_DTYPES = {"x": "bfloat16", "w_mat": "bfloat16"}
WORLD_SIZE = 8

def _to_bf16(x):
    u = np.ascontiguousarray(x, dtype=np.float32).view(np.uint32)
    return ((u + 0x7FFF + ((u >> 16) & 1)) & 0xFFFF0000).view(np.float32)

def setup_inputs():

    x = _to_bf16(np.random.randn(4096, 4096).astype(np.float32))
    w_mat = _to_bf16((np.random.randn(4096, 16384) / np.sqrt(4096)).astype(np.float32))
    return {"x": x, "w_mat": w_mat}

def _epilogue(y):
    return y

def reference(x, w_mat):
    y = (x @ w_mat).astype(np.float32)
    return _epilogue(y)
```

```cpp
#include <hip/hip_runtime.h>
#include <rccl/rccl.h>
#include <cstdio>
#include <type_traits>

namespace {
typedef __bf16 bf16x2 __attribute__((ext_vector_type(2)));
typedef __bf16 bf16x4 __attribute__((ext_vector_type(4)));
typedef __bf16 bf16x8 __attribute__((ext_vector_type(8)));
typedef float f32x2 __attribute__((ext_vector_type(2)));
typedef float f32x4 __attribute__((ext_vector_type(4)));
typedef unsigned u32x4 __attribute__((ext_vector_type(4)));
#define LAS __attribute__((address_space(3)))

constexpr int NR = 8, MR = 512, KD = 4096, ND = 16384, NS = 2048;
constexpr int BK = 64, NKT = KD / BK, BN = 128;
#define H1 10
constexpr int HB = 16 - H1;
constexpr int H2 = HB + 1, HL = 18 - H1;
constexpr int BM1 = 16 * H1, RA = 2 * BM1, RB = MR - RA;
constexpr int BBYTES = BK * BN * 2;
constexpr int NST = 4;
constexpr int STG_OFF = NST * (H2 * 2048 + BBYTES);
constexpr int CTL_OFF = NST * (H1 * 2048 + BBYTES);
constexpr int LDS_BYTES = 160 * 1024;
static_assert(H1 >= HL && H1 >= H2 && CTL_OFF + 16 <= LDS_BYTES && STG_OFF + H1 * 4096 <= LDS_BYTES, "lds");
constexpr int NCOMM = 16, GEMM0 = 16;
#define SLOT 80
#define TSHIFT 300

constexpr size_t WS_CALL = 0;
constexpr size_t WS_XCNT = 32768;
constexpr size_t WS_OCNT = 36864;
constexpr size_t WS_TMO = 40960;
constexpr size_t WS_TST = 45056;
constexpr size_t WS_DBG = 131072;
constexpr size_t WS_X = 1u << 20;
constexpr size_t XSLOT = (size_t)RB * KD * 2;
constexpr size_t WS_NEED = WS_X + 2 * NR * XSLOT;
constexpr unsigned long long DEADLINE = 40000000ull;

struct Params {
  const char* x; const char* w;
  char* out[NR]; char* ws[NR];
  int rank;
};

#define WAIT_V(n) asm volatile("s_waitcnt vmcnt(%0)" ::"n"(n) : "memory")
#define WAIT_L0() __builtin_amdgcn_s_waitcnt(0xC07F)
#define CFENCE() asm volatile("" ::: "memory")
#define BAR() do { CFENCE(); __builtin_amdgcn_s_barrier(); CFENCE(); } while (0)

__device__ __forceinline__ unsigned pk(float a, float b) { f32x2 v = {a, b}; return __builtin_bit_cast(unsigned, __builtin_convertvector(v, bf16x2)); }
typedef __amdgpu_buffer_rsrc_t rsrc_t;
__device__ __forceinline__ rsrc_t mk_rsrc(const void* base, unsigned bytes) {
  const unsigned long long b = (unsigned long long)base;
  const unsigned lo = __builtin_amdgcn_readfirstlane((unsigned)b), hi = __builtin_amdgcn_readfirstlane((unsigned)(b >> 32));
  return __builtin_amdgcn_make_buffer_rsrc((void*)(((unsigned long long)hi << 32) | lo), 0, bytes, 0x00020000);
}
constexpr int AUX_SYS = 17, AUX_DEV = 16;
__device__ __forceinline__ void pace(int& s, int smax) {
  unsigned t0;
  asm volatile(
      "1:\n\t"
      "s_getreg_b32 %1, hwreg(HW_REG_IB_STS)\n\t"
      "s_and_b32 %1, %1, 0xc0000f\n\t"
      "s_cbranch_scc0 3f\n\t"
      "s_cmp_ge_i32 %0, %2\n\t"
      "s_cbranch_scc1 2f\n\t"
      "s_barrier\n\t"
      "s_add_i32 %0, %0, 1\n\t"
      "s_branch 1b\n\t"
      "2:\n\t"
      "s_sleep 1\n\t"
      "s_branch 1b\n\t"
      "3:"
      : "+s"(s), "=&s"(t0) : "s"(smax) : "memory", "scc");
}
#define STAMP(i) do { } while (0)

__device__ __forceinline__ bool wait_ge(unsigned* f, bool skip, unsigned want, unsigned* tmo, unsigned call) {
  const unsigned long long t0 = __builtin_amdgcn_s_memrealtime();
  for (;;) {
    const unsigned v = __hip_atomic_load(f, __ATOMIC_RELAXED, __HIP_MEMORY_SCOPE_SYSTEM);
    if (__all(skip || v >= want)) return true;
    if (__builtin_amdgcn_readfirstlane(__hip_atomic_load(tmo, __ATOMIC_RELAXED, __HIP_MEMORY_SCOPE_AGENT)) == call ||
        __builtin_amdgcn_s_memrealtime() - t0 > DEADLINE) {
      __hip_atomic_store(tmo, call, __ATOMIC_RELAXED, __HIP_MEMORY_SCOPE_AGENT);
      return false;
    }
    __builtin_amdgcn_s_sleep(2);
  }
}

template <int MT> struct Frag { bf16x8 a[MT]; bf16x8 b[4]; };

template <int MT>
__device__ __forceinline__ void rd(Frag<MT>& f, LAS char* base, const int (&ab)[2], const int (&bb)[4], int ks) {
#pragma unroll
  for (int n = 0; n < 4; ++n) {
    bf16x4 lo = __builtin_amdgcn_ds_read_tr16_b64_v4bf16((LAS bf16x4*)(base + bb[n] + (ks * 32) * 256));
    bf16x4 hi = __builtin_amdgcn_ds_read_tr16_b64_v4bf16((LAS bf16x4*)(base + bb[n] + (ks * 32 + 4) * 256));
    f.b[n] = __builtin_shufflevector(lo, hi, 0, 1, 2, 3, 4, 5, 6, 7);
  }
#pragma unroll
  for (int m = 0; m < MT; ++m) f.a[m] = *(LAS bf16x8*)(base + ab[ks] + m * 2048);
}
template <int MT>
__device__ __forceinline__ void mm(f32x4 (&acc)[MT][4], const Frag<MT>& f) {
#pragma unroll
  for (int m = 0; m < MT; ++m)
#pragma unroll
    for (int n = 0; n < 4; ++n) acc[m][n] = __builtin_amdgcn_mfma_f32_16x16x32_bf16(f.b[n], f.a[m], acc[m][n], 0, 0, 0);
}
template <int NMF, int NRD>
__device__ __forceinline__ void sched() {
  constexpr int NI = NRD < NMF ? NRD : NMF;
  if constexpr (NRD > NMF) __builtin_amdgcn_sched_group_barrier(0x100, NRD - NMF, 0);
#pragma unroll
  for (int i = 0; i < NI; ++i) { __builtin_amdgcn_sched_group_barrier(0x8, 1, 0); __builtin_amdgcn_sched_group_barrier(0x100, 1, 0); }
  if constexpr (NMF > NI) __builtin_amdgcn_sched_group_barrier(0x8, NMF - NI, 0);
}
__device__ __forceinline__ u32x4 pack8(const f32x4& c0, const f32x4& c1) {
  unsigned x0 = pk(c0[0], c0[1]), x1 = pk(c0[2], c0[3]), y0 = pk(c1[0], c1[1]), y1 = pk(c1[2], c1[3]);
  auto s0 = __builtin_amdgcn_permlane16_swap(x0, y0, false, false);
  auto s1 = __builtin_amdgcn_permlane16_swap(x1, y1, false, false);
  return u32x4{s0[0], s1[0], s0[1], s1[1]};
}

template <int MT>
__device__ __forceinline__ void mfma_tile(f32x4 (&acc)[MT][4], LAS char* lds, int H, int s0, int wc, int lane, LAS unsigned* ctl, unsigned* tmo, unsigned& word) {
  const int fr = lane & 15, fq = lane >> 4;
  const int stb = H * 2048 + BBYTES;
  int ab[2], bb[4];
#pragma unroll
  for (int ks = 0; ks < 2; ++ks) ab[ks] = (s0 * 16 + fr) * 128 + (((ks * 4 + fq) ^ ((fr >> 1) & 7)) << 4);
  {
    const int q = (lane & 15) >> 2, p = lane & 3, k = 8 * fq + q;
    const int sw = (k & 3) | (((k >> 3) & 1) << 2);
#pragma unroll
    for (int n = 0; n < 4; ++n) bb[n] = H * 2048 + k * 256 + (((wc * 4 + n) ^ sw) << 5) + p * 8;
  }
#pragma unroll
  for (int m = 0; m < MT; ++m)
#pragma unroll
    for (int n = 0; n < 4; ++n) acc[m][n] = f32x4{0.f, 0.f, 0.f, 0.f};
  Frag<MT> f0, f1;
  BAR();
  if (ctl) word = ctl[0]; else word = __hip_atomic_load(tmo, __ATOMIC_RELAXED, __HIP_MEMORY_SCOPE_AGENT);
  rd<MT>(f0, lds, ab, bb, 0);
  int so = 0;
  for (int t = 0; t < NKT - 1; ++t) {
    rd<MT>(f1, lds + so, ab, bb, 1);
    mm<MT>(acc, f0);
    sched<MT * 4, MT + 8>();
    WAIT_L0();
    BAR();
    so = (so + stb == NST * stb) ? 0 : so + stb;
    rd<MT>(f0, lds + so, ab, bb, 0);
    mm<MT>(acc, f1);
    sched<MT * 4, MT + 8>();
  }
  rd<MT>(f1, lds + so, ab, bb, 1);
  mm<MT>(acc, f0);
  sched<MT * 4, MT + 8>();
  WAIT_L0();
  BAR();
  mm<MT>(acc, f1);
}

template <int H, int HA>
__device__ __forceinline__ void load_tile(LAS char* lds, const char* a, const char* a2, const char* b, int lw, int lane, LAS unsigned* ctl, unsigned& call) {
  constexpr int STB = H * 2048 + BBYTES, D = H + 8;
  const int arow = lw * 8 + (lane >> 3);
  const unsigned va = (unsigned)(arow * (KD * 2) + (((lane & 7) ^ ((arow >> 1) & 7)) << 4));
  unsigned vb[2];
#pragma unroll
  for (int i = 0; i < 2; ++i) {
    const int k = (lw + 2 * i) * 4 + (lane >> 4);
    const int sw = (k & 3) | (((k >> 3) & 1) << 2);
    vb[i] = (unsigned)(k * (ND * 2) + (((lane & 15) ^ (sw << 1)) << 4));
  }
  auto issue = [&](int t) {
    LAS char* dst = lds + (t & 3) * STB + lw * 1024;
    const char* ga = a + t * 128;
    const char* ga2 = a2 + t * 128;
    const char* gb = b + (size_t)t * BK * ND * 2;
#pragma unroll
    for (int i = 0; i < 8; ++i) __builtin_amdgcn_global_load_lds((const unsigned*)(gb + (size_t)(i >> 1) * 16 * ND * 2 + vb[i & 1]), (LAS unsigned*)(dst + H * 2048 + i * 2048), 16, 0, 0);
#pragma unroll
    for (int i = 0; i < H; ++i) __builtin_amdgcn_global_load_lds((const unsigned*)((i < HA ? ga + (size_t)i * 16 * KD * 2 : ga2 + (size_t)(i - HA) * 16 * KD * 2) + va), (LAS unsigned*)(dst + i * 2048), 16, 0, 0);
  };
#pragma unroll
  for (int t = 0; t < NST; ++t) issue(t);
  WAIT_V(3 * D); BAR();
  if (ctl) call = __builtin_amdgcn_readfirstlane(ctl[0]);
  for (int t = 0; t < NKT - NST; ++t) { WAIT_V(2 * D); BAR(); issue(t + NST); }
  WAIT_V(2 * D); BAR();
  WAIT_V(D); BAR();
  WAIT_V(0); BAR();
  BAR();
}

__global__ __launch_bounds__(512, 2) void gemm_a2a(Params P) {
  __shared__ __attribute__((aligned(1024))) char smem[LDS_BYTES];
  LAS char* lds = (LAS char*)smem;
  const int tid = threadIdx.x, lane = tid & 63;
  const int wid = __builtin_amdgcn_readfirstlane(tid >> 6);
  const int rank = P.rank;
  char* const myws = P.ws[rank];
  unsigned* const tmo = (unsigned*)(myws + WS_TMO);
  LAS unsigned* ctl = (LAS unsigned*)(lds + CTL_OFF);
  const int bid = blockIdx.x;

  if (bid < GEMM0) {
    constexpr int NPC = (int)(XSLOT / 4096);
    const rsrc_t sr = mk_rsrc(P.x + (size_t)RA * KD * 2, XSLOT);
    const int gw = bid * 8 + wid;
    u32x4 v[4], vn[4];
#pragma unroll
    for (int i = 0; i < 4; ++i) vn[i] = __builtin_amdgcn_raw_buffer_load_b128(sr, lane * 16 + i * 1024, gw * 4096, 0);
    unsigned call = 0;
    if (lane == 0) call = __hip_atomic_fetch_add((unsigned*)(myws + WS_CALL + bid * 64 + wid * 4), 1u, __ATOMIC_RELAXED, __HIP_MEMORY_SCOPE_AGENT) + 1;
    call = __builtin_amdgcn_readfirstlane(call);
    STAMP(0);
    const unsigned t0 = (unsigned)__builtin_amdgcn_s_memrealtime();
    if (tid == 0) ctl[0] = 0;
    WAIT_L0();
    BAR();
    for (int pc = gw; pc < NPC; pc += NCOMM * 8) {
#pragma unroll
      for (int i = 0; i < 4; ++i) v[i] = vn[i];
#pragma unroll
      for (int i = 0; i < 4; ++i) vn[i] = __builtin_amdgcn_raw_buffer_load_b128(sr, lane * 16 + i * 1024, (pc + NCOMM * 8) * 4096, 0);
      for (int e = 0; e < 7; ++e) {
        const unsigned pp = (unsigned)(rank + 1 + (e + gw) % 7) & 7u;
        const rsrc_t dr = mk_rsrc(P.ws[pp] + WS_X + ((call & 1) * NR + rank) * XSLOT, XSLOT);
#pragma unroll
        for (int i = 0; i < 4; ++i) __builtin_amdgcn_raw_buffer_store_b128(v[i], dr, lane * 16 + i * 1024, pc * 4096, AUX_SYS);
      }
      if (bid == 0 && lane == 0) {
        constexpr int NPW = NPC / NCOMM, NQ = NPW * 2 / 3;
        if (__hip_atomic_fetch_add(ctl, 1u, __ATOMIC_RELAXED, __HIP_MEMORY_SCOPE_WORKGROUP) == NQ - 1) {
          const unsigned te = t0 + ((unsigned)__builtin_amdgcn_s_memrealtime() - t0) * NPW / NQ + TSHIFT;
          __hip_atomic_store((unsigned long long*)(myws + WS_TST), ((unsigned long long)call << 32) | te, __ATOMIC_RELAXED, __HIP_MEMORY_SCOPE_AGENT);
        }
      }
    }
    WAIT_V(0);
    BAR();
    if (wid != 0) return;
    if (lane < 7) __hip_atomic_fetch_add((unsigned*)(P.ws[(rank + 1 + lane) & 7] + WS_XCNT + rank * 128), 1u, __ATOMIC_RELAXED, __HIP_MEMORY_SCOPE_SYSTEM);
    STAMP(5);
    if (bid == 0) {
      if (!wait_ge((unsigned*)(myws + WS_OCNT + (lane & 7) * 128), (lane & 7) == rank, call * 32u, tmo, call)) {
        for (int i = 0; i < 20000; ++i) __builtin_amdgcn_s_sleep(127);
        if (lane == 0) *(unsigned short*)(P.out[rank] + (size_t)rank * MR * NS * 2) = 0x7fc0;
      }
      __builtin_amdgcn_fence(__ATOMIC_ACQUIRE, "");
      STAMP(9);
    }
    return;
  }

  const int g = bid & 7, idx = (bid >> 3) - 2;
  const bool local = idx >= 28;
  const int i1 = local ? 0 : (idx & 1), cl = idx >> 1;
  const int d1 = cl % 7;
  const unsigned p1 = local ? (unsigned)rank : ((unsigned)(rank + 1 + d1) & 7u);
  const int c1 = g * 2 + (local ? idx - 28 : cl / 7);
  const int mt2 = (g & 1) * 7 + idx % 7;
  const int c2 = local ? c1 : (g >> 1) * 4 + idx / 7;
  const unsigned q2 = (unsigned)(rank + 1 + (mt2 >> 1)) & 7u;
  const int i2 = mt2 & 1;
  const int own2 = 18 * 16 + mt2 * 16;

  unsigned call = 0;
  if (wid == 7) return;
  if (wid == 6) {
    if (lane == 0) call = __hip_atomic_fetch_add((unsigned*)(myws + WS_CALL + bid * 64), 1u, __ATOMIC_RELAXED, __HIP_MEMORY_SCOPE_AGENT) + 1;
    call = __builtin_amdgcn_readfirstlane(call);
    if (lane == 0) ctl[0] = call;
    WAIT_L0();
    BAR();
    if (local) return;
    constexpr int SPARK = NKT + 1, SLAST = 2 * NKT + 3;
    int s = 0;
    const int ord = (c1 & 1) * 16 + i1 * 8 + ((g + d1) & 7);
    const unsigned long long* pw = (const unsigned long long*)(myws + ((lane & 1) ? WS_TST : WS_XCNT + q2 * 128));
    bool xok = false, tok = false, pushed = false, ok = true;
    unsigned slot = 0;
    auto look = [&](int smax) {
      unsigned long long v;
      asm volatile("global_load_dwordx2 %0, %1, off sc0 sc1" : "=v"(v) : "v"(pw) : "memory");
      pace(s, smax);
      asm volatile("s_waitcnt vmcnt(0)" : "+v"(v)::"memory");
      xok = xok || __builtin_amdgcn_readlane((unsigned)v, 0) >= call * (unsigned)NCOMM;
      if (!tok && __builtin_amdgcn_readlane((unsigned)(v >> 32), 1) == call) { tok = true; slot = __builtin_amdgcn_readlane((unsigned)v, 1) + ord * SLOT; }
    };
    auto push = [&]() {
      STAMP(6);
      const rsrc_t dr = mk_rsrc(P.out[p1] + ((size_t)(rank * MR + i1 * BM1) * NS + c1 * BN) * 2, (BM1 - 1) * NS * 2 + 256);
#pragma unroll
      for (int h = 0; h < 2; ++h) {
        const int r0 = h * (BM1 / 2) + (lane >> 4);
        u32x4 v[H1 * 2];
#pragma unroll
        for (int i = 0; i < H1 * 2; ++i) { const int row = r0 + i * 4; v[i] = *(LAS u32x4*)(lds + STG_OFF + row * 256 + (((lane & 15) ^ (row & 7)) << 4)); }
#pragma unroll
        for (int i = 0; i < H1 * 2; ++i) __builtin_amdgcn_raw_buffer_store_b128(v[i], dr, r0 * (NS * 2) + (lane & 15) * 16, i * 4 * NS * 2, AUX_SYS);
      }
      pushed = true;
    };
    while (s < SPARK - 8) { BAR(); ++s; }
    while (s < SPARK - 1 && !(xok && tok)) { look(SPARK - 1); if (s < SPARK - 1) { BAR(); ++s; } }
    while (s < SPARK) { BAR(); ++s; }
    {
      const unsigned long long t0 = __builtin_amdgcn_s_memrealtime();
      while (!xok) {
        if (!pushed && tok && (int)((unsigned)__builtin_amdgcn_s_memrealtime() - slot) >= 0) push();
        look(SPARK);
        if (xok) break;
        if (__builtin_amdgcn_readfirstlane(__hip_atomic_load(tmo, __ATOMIC_RELAXED, __HIP_MEMORY_SCOPE_AGENT)) == call ||
            __builtin_amdgcn_s_memrealtime() - t0 > DEADLINE) {
          __hip_atomic_store(tmo, call, __ATOMIC_RELAXED, __HIP_MEMORY_SCOPE_AGENT);
          break;
        }
      }
      __builtin_amdgcn_fence(__ATOMIC_ACQUIRE, "");
      WAIT_V(0);
      STAMP(8);
      BAR(); ++s;
      while (!pushed) {
        if (!tok) {
          look(SLAST);
          if (!tok && s >= SLAST && __builtin_amdgcn_s_memrealtime() - t0 > DEADLINE) { tok = true; slot = (unsigned)__builtin_amdgcn_s_memrealtime(); }
        }
        if (tok && (int)((unsigned)__builtin_amdgcn_s_memrealtime() - slot) >= 0) { push(); break; }
        if (s < SLAST) { BAR(); ++s; } else __builtin_amdgcn_s_sleep(1);
      }
    }
    pace(s, SLAST);
    WAIT_V(0);
    if (ok && lane == 0) __hip_atomic_fetch_add((unsigned*)(P.ws[p1] + WS_OCNT + rank * 128), 1u, __ATOMIC_RELAXED, __HIP_MEMORY_SCOPE_SYSTEM);
    STAMP(7);
    return;
  }
  if (wid >= 4) {
    const int lw = wid - 4;
    load_tile<H1, H1>(lds, P.x + (size_t)i1 * BM1 * KD * 2, P.x, P.w + (size_t)(p1 * NS + c1 * BN) * 2, lw, lane, ctl, call);
    const char* b2 = P.w + (size_t)(rank * NS + c2 * BN) * 2;
    if (local) {
      load_tile<HL, HL>(lds, P.x + (size_t)BM1 * KD * 2, P.x, b2, lw, lane, nullptr, call);
    } else {
      BAR();
      BAR();
      load_tile<H2, HB>(lds, myws + WS_X + ((call & 1) * NR + q2) * XSLOT + (size_t)i2 * HB * 16 * KD * 2, P.x + (size_t)own2 * KD * 2, b2, lw, lane, nullptr, call);
    }
    return;
  }
  const int wr = wid >> 1, wc = wid & 1, fr = lane & 15, fq = lane >> 4;
  const int coff = (wc * 64 + (fq & 1) * 16 + (fq >> 1) * 8) * 2;
  char* const o = P.out[rank];
  {
    constexpr int MT = H1 / 2;
    f32x4 acc[MT][4];
    STAMP(0);
    mfma_tile<MT>(acc, lds, H1, wr * MT, wc, lane, ctl, tmo, call);
    STAMP(1);
    if (local) {
      char* dst = o + ((size_t)(rank * MR + wr * MT * 16 + fr) * NS + c1 * BN) * 2 + coff;
#pragma unroll
      for (int m = 0; m < MT; ++m)
#pragma unroll
        for (int u = 0; u < 2; ++u) *(u32x4*)(dst + (size_t)m * 16 * NS * 2 + u * 64) = pack8(acc[m][2 * u], acc[m][2 * u + 1]);
    } else {
#pragma unroll
      for (int m = 0; m < MT; ++m)
#pragma unroll
        for (int u = 0; u < 2; ++u)
          *(LAS u32x4*)(lds + STG_OFF + (wr * MT * 16 + m * 16 + fr) * 256 + ((((coff >> 4) + u * 4) ^ (fr & 7)) << 4)) = pack8(acc[m][2 * u], acc[m][2 * u + 1]);
      WAIT_L0();
      BAR();
      BAR();
    }
    STAMP(2);
  }
  auto round2 = [&](auto mtc, int H, int s0) {
    constexpr int MT = decltype(mtc)::value;
    f32x4 acc[MT][4];
    unsigned tw;
    mfma_tile<MT>(acc, lds, H, s0, wc, lane, nullptr, tmo, tw);
    STAMP(3);
    const bool bad = tw == call;
#pragma unroll
    for (int m = 0; m < MT; ++m) {
      const int s = s0 + m;
      const int row = local ? rank * MR + BM1 + s * 16 : (s < HB ? (int)q2 * MR + RA + (i2 * HB + s) * 16 : rank * MR + own2);
      char* dst = o + ((size_t)(row + fr) * NS + c2 * BN) * 2 + coff;
#pragma unroll
      for (int u = 0; u < 2; ++u) {
        u32x4 v = pack8(acc[m][2 * u], acc[m][2 * u + 1]);
        if (bad) v = u32x4{0x7fc07fc0u, 0x7fc07fc0u, 0x7fc07fc0u, 0x7fc07fc0u};
        *(u32x4*)(dst + u * 64) = v;
      }
    }
    STAMP(4);
  };
  constexpr int MTU = (HL + 1) / 2;
  static_assert(HL % 2 == 0 && H2 == HL - 1, "round-2 split");
  if (wr == 0 || local) round2(std::integral_constant<int, MTU>{}, local ? HL : H2, wr * MTU);
  else round2(std::integral_constant<int, MTU - 1>{}, H2, MTU);
}
}

extern "C" void kernel_launch(int rank, int nranks, ncclComm_t comm, hipStream_t stream,
                              void* const* d_in, const int* in_sizes, int n_in,
                              void* d_out, int out_size, const int* in_dtypes, int out_dtype,
                              void* const* d_out_all, void* const* d_ws_all, size_t ws_bytes) {
  if (nranks != NR || (unsigned)rank >= (unsigned)NR || n_in < 2 || ws_bytes < WS_NEED ||
      in_sizes[0] != MR * KD || in_sizes[1] != KD * ND || out_size != NR * MR * NS) {
    fprintf(stderr, "kernel_launch: unexpected problem shape\n");
    return;
  }
  Params P;
  P.x = (const char*)d_in[0]; P.w = (const char*)d_in[1]; P.rank = rank;
  for (int p = 0; p < NR; ++p) { P.out[p] = (char*)d_out_all[p]; P.ws[p] = (char*)d_ws_all[p]; }
  gemm_a2a<<<256, 512, 0, stream>>>(P);
}
```

```cpp
#include <hip/hip_runtime.h>
#include <rccl/rccl.h>
#include <cstdio>
#include <type_traits>

namespace {
typedef __bf16 bf16x2 __attribute__((ext_vector_type(2)));
typedef __bf16 bf16x4 __attribute__((ext_vector_type(4)));
typedef __bf16 bf16x8 __attribute__((ext_vector_type(8)));
typedef float f32x2 __attribute__((ext_vector_type(2)));
typedef float f32x4 __attribute__((ext_vector_type(4)));
typedef unsigned u32x4 __attribute__((ext_vector_type(4)));
#define LAS __attribute__((address_space(3)))

constexpr int NR = 8, MR = 512, KD = 4096, ND = 16384, NS = 2048;
constexpr int BK = 64, NKT = KD / BK, BN = 128;
#define H1 10
constexpr int HB = 16 - H1;
constexpr int H2 = HB + 1, HL = 18 - H1;
constexpr int BM1 = 16 * H1, RA = 2 * BM1, RB = MR - RA;
constexpr int BBYTES = BK * BN * 2;
constexpr int NST = 4;
constexpr int STG_OFF = NST * (H2 * 2048 + BBYTES);
constexpr int CTL_OFF = NST * (H1 * 2048 + BBYTES);
constexpr int LDS_BYTES = 160 * 1024;
static_assert(H1 >= HL && H1 >= H2 && CTL_OFF + 16 <= LDS_BYTES && STG_OFF + H1 * 4096 <= LDS_BYTES, "lds");
constexpr int NCOMM = 16, GEMM0 = 16;
#define SLOT 80
#define AUXW1 0
#define TSHIFT 100

constexpr size_t WS_CALL = 0;
constexpr size_t WS_XCNT = 32768;
constexpr size_t WS_OCNT = 36864;
constexpr size_t WS_TMO = 40960;
constexpr size_t WS_TST = 45056;
constexpr size_t WS_DBG = 131072;
constexpr size_t WS_X = 1u << 20;
constexpr size_t XSLOT = (size_t)RB * KD * 2;
constexpr size_t WS_NEED = WS_X + 2 * NR * XSLOT;
constexpr unsigned long long DEADLINE = 40000000ull;

struct Params {
  const char* x; const char* w;
  char* out[NR]; char* ws[NR];
  int rank;
};

#define WAIT_V(n) asm volatile("s_waitcnt vmcnt(%0)" ::"n"(n) : "memory")
#define WAIT_L0() __builtin_amdgcn_s_waitcnt(0xC07F)
#define CFENCE() asm volatile("" ::: "memory")
#define BAR() do { CFENCE(); __builtin_amdgcn_s_barrier(); CFENCE(); } while (0)

__device__ __forceinline__ unsigned pk(float a, float b) { f32x2 v = {a, b}; return __builtin_bit_cast(unsigned, __builtin_convertvector(v, bf16x2)); }
typedef __amdgpu_buffer_rsrc_t rsrc_t;
__device__ __forceinline__ rsrc_t mk_rsrc(const void* base, unsigned bytes) {
  const unsigned long long b = (unsigned long long)base;
  const unsigned lo = __builtin_amdgcn_readfirstlane((unsigned)b), hi = __builtin_amdgcn_readfirstlane((unsigned)(b >> 32));
  return __builtin_amdgcn_make_buffer_rsrc((void*)(((unsigned long long)hi << 32) | lo), 0, bytes, 0x00020000);
}
constexpr int AUX_SYS = 17, AUX_DEV = 16;
__device__ __forceinline__ void pace(int& s, int smax) {
  unsigned t0;
  asm volatile(
      "1:\n\t"
      "s_getreg_b32 %1, hwreg(HW_REG_IB_STS)\n\t"
      "s_and_b32 %1, %1, 0xc0000f\n\t"
      "s_cbranch_scc0 3f\n\t"
      "s_cmp_ge_i32 %0, %2\n\t"
      "s_cbranch_scc1 2f\n\t"
      "s_barrier\n\t"
      "s_add_i32 %0, %0, 1\n\t"
      "s_branch 1b\n\t"
      "2:\n\t"
      "s_sleep 1\n\t"
      "s_branch 1b\n\t"
      "3:"
      : "+s"(s), "=&s"(t0) : "s"(smax) : "memory", "scc");
}
#define STAMP(i) do { } while (0)

__device__ __forceinline__ bool wait_ge(unsigned* f, bool skip, unsigned want, unsigned* tmo, unsigned call) {
  const unsigned long long t0 = __builtin_amdgcn_s_memrealtime();
  for (;;) {
    const unsigned v = __hip_atomic_load(f, __ATOMIC_RELAXED, __HIP_MEMORY_SCOPE_SYSTEM);
    if (__all(skip || v >= want)) return true;
    if (__builtin_amdgcn_readfirstlane(__hip_atomic_load(tmo, __ATOMIC_RELAXED, __HIP_MEMORY_SCOPE_AGENT)) == call ||
        __builtin_amdgcn_s_memrealtime() - t0 > DEADLINE) {
      __hip_atomic_store(tmo, call, __ATOMIC_RELAXED, __HIP_MEMORY_SCOPE_AGENT);
      return false;
    }
    __builtin_amdgcn_s_sleep(2);
  }
}

template <int MT> struct Frag { bf16x8 a[MT]; bf16x8 b[4]; };

template <int MT>
__device__ __forceinline__ void rd(Frag<MT>& f, LAS char* base, const int (&ab)[2], const int (&bb)[4], int ks) {
#pragma unroll
  for (int n = 0; n < 4; ++n) {
    bf16x4 lo = __builtin_amdgcn_ds_read_tr16_b64_v4bf16((LAS bf16x4*)(base + bb[n] + (ks * 32) * 256));
    bf16x4 hi = __builtin_amdgcn_ds_read_tr16_b64_v4bf16((LAS bf16x4*)(base + bb[n] + (ks * 32 + 4) * 256));
    f.b[n] = __builtin_shufflevector(lo, hi, 0, 1, 2, 3, 4, 5, 6, 7);
  }
#pragma unroll
  for (int m = 0; m < MT; ++m) f.a[m] = *(LAS bf16x8*)(base + ab[ks] + m * 2048);
}
template <int MT>
__device__ __forceinline__ void mm(f32x4 (&acc)[MT][4], const Frag<MT>& f) {
#pragma unroll
  for (int m = 0; m < MT; ++m)
#pragma unroll
    for (int n = 0; n < 4; ++n) acc[m][n] = __builtin_amdgcn_mfma_f32_16x16x32_bf16(f.b[n], f.a[m], acc[m][n], 0, 0, 0);
}
template <int NMF, int NRD>
__device__ __forceinline__ void sched() {
  constexpr int NI = NRD < NMF ? NRD : NMF;
  if constexpr (NRD > NMF) __builtin_amdgcn_sched_group_barrier(0x100, NRD - NMF, 0);
#pragma unroll
  for (int i = 0; i < NI; ++i) { __builtin_amdgcn_sched_group_barrier(0x8, 1, 0); __builtin_amdgcn_sched_group_barrier(0x100, 1, 0); }
  if constexpr (NMF > NI) __builtin_amdgcn_sched_group_barrier(0x8, NMF - NI, 0);
}
__device__ __forceinline__ u32x4 pack8(const f32x4& c0, const f32x4& c1) {
  unsigned x0 = pk(c0[0], c0[1]), x1 = pk(c0[2], c0[3]), y0 = pk(c1[0], c1[1]), y1 = pk(c1[2], c1[3]);
  auto s0 = __builtin_amdgcn_permlane16_swap(x0, y0, false, false);
  auto s1 = __builtin_amdgcn_permlane16_swap(x1, y1, false, false);
  return u32x4{s0[0], s1[0], s0[1], s1[1]};
}

template <int MT>
__device__ __forceinline__ void mfma_tile(f32x4 (&acc)[MT][4], LAS char* lds, int H, int s0, int wc, int lane, LAS unsigned* ctl, unsigned* tmo, unsigned& word) {
  const int fr = lane & 15, fq = lane >> 4;
  const int stb = H * 2048 + BBYTES;
  int ab[2], bb[4];
#pragma unroll
  for (int ks = 0; ks < 2; ++ks) ab[ks] = (s0 * 16 + fr) * 128 + (((ks * 4 + fq) ^ ((fr >> 1) & 7)) << 4);
  {
    const int q = (lane & 15) >> 2, p = lane & 3, k = 8 * fq + q;
    const int sw = (k & 3) | (((k >> 3) & 1) << 2);
#pragma unroll
    for (int n = 0; n < 4; ++n) bb[n] = H * 2048 + k * 256 + (((wc * 4 + n) ^ sw) << 5) + p * 8;
  }
#pragma unroll
  for (int m = 0; m < MT; ++m)
#pragma unroll
    for (int n = 0; n < 4; ++n) acc[m][n] = f32x4{0.f, 0.f, 0.f, 0.f};
  Frag<MT> f0, f1;
  BAR();
  if (ctl) word = ctl[0]; else word = __hip_atomic_load(tmo, __ATOMIC_RELAXED, __HIP_MEMORY_SCOPE_AGENT);
  rd<MT>(f0, lds, ab, bb, 0);
  int so = 0;
  for (int t = 0; t < NKT - 1; ++t) {
    rd<MT>(f1, lds + so, ab, bb, 1);
    mm<MT>(acc, f0);
    sched<MT * 4, MT + 8>();
    WAIT_L0();
    BAR();
    so = (so + stb == NST * stb) ? 0 : so + stb;
    rd<MT>(f0, lds + so, ab, bb, 0);
    mm<MT>(acc, f1);
    sched<MT * 4, MT + 8>();
  }
  rd<MT>(f1, lds + so, ab, bb, 1);
  mm<MT>(acc, f0);
  sched<MT * 4, MT + 8>();
  WAIT_L0();
  BAR();
  mm<MT>(acc, f1);
}

template <int H, int HA, int AUXW>
__device__ __forceinline__ void load_tile(LAS char* lds, const char* a, const char* a2, const char* b, int lw, int lane, LAS unsigned* ctl, unsigned& call) {
  constexpr int STB = H * 2048 + BBYTES, D = H + 8;
  const int arow = lw * 8 + (lane >> 3);
  const unsigned va = (unsigned)(arow * (KD * 2) + (((lane & 7) ^ ((arow >> 1) & 7)) << 4));
  unsigned vb[2];
#pragma unroll
  for (int i = 0; i < 2; ++i) {
    const int k = (lw + 2 * i) * 4 + (lane >> 4);
    const int sw = (k & 3) | (((k >> 3) & 1) << 2);
    vb[i] = (unsigned)(k * (ND * 2) + (((lane & 15) ^ (sw << 1)) << 4));
  }
  auto issue = [&](int t) {
    LAS char* dst = lds + (t & 3) * STB + lw * 1024;
    const char* ga = a + t * 128;
    const char* ga2 = a2 + t * 128;
    const char* gb = b + (size_t)t * BK * ND * 2;
#pragma unroll
    for (int i = 0; i < 8; ++i) __builtin_amdgcn_global_load_lds((const unsigned*)(gb + (size_t)(i >> 1) * 16 * ND * 2 + vb[i & 1]), (LAS unsigned*)(dst + H * 2048 + i * 2048), 16, 0, AUXW);
#pragma unroll
    for (int i = 0; i < H; ++i) __builtin_amdgcn_global_load_lds((const unsigned*)((i < HA ? ga + (size_t)i * 16 * KD * 2 : ga2 + (size_t)(i - HA) * 16 * KD * 2) + va), (LAS unsigned*)(dst + i * 2048), 16, 0, 0);
  };
#pragma unroll
  for (int t = 0; t < NST; ++t) issue(t);
  WAIT_V(3 * D); BAR();
  if (ctl) call = __builtin_amdgcn_readfirstlane(ctl[0]);
  for (int t = 0; t < NKT - NST; ++t) { WAIT_V(2 * D); BAR(); issue(t + NST); }
  WAIT_V(2 * D); BAR();
  WAIT_V(D); BAR();
  WAIT_V(0); BAR();
  BAR();
}

__global__ __launch_bounds__(512, 2) void gemm_a2a(Params P) {
  __shared__ __attribute__((aligned(1024))) char smem[LDS_BYTES];
  LAS char* lds = (LAS char*)smem;
  const int tid = threadIdx.x, lane = tid & 63;
  const int wid = __builtin_amdgcn_readfirstlane(tid >> 6);
  const int rank = P.rank;
  char* const myws = P.ws[rank];
  unsigned* const tmo = (unsigned*)(myws + WS_TMO);
  LAS unsigned* ctl = (LAS unsigned*)(lds + CTL_OFF);
  const int bid = blockIdx.x;

  if (bid < GEMM0) {
    constexpr int NPC = (int)(XSLOT / 4096);
    const rsrc_t sr = mk_rsrc(P.x + (size_t)RA * KD * 2, XSLOT);
    const int gw = bid * 8 + wid;
    u32x4 v[4], vn[4];
#pragma unroll
    for (int i = 0; i < 4; ++i) vn[i] = __builtin_amdgcn_raw_buffer_load_b128(sr, lane * 16 + i * 1024, gw * 4096, 0);
    unsigned call = 0;
    if (lane == 0) call = __hip_atomic_fetch_add((unsigned*)(myws + WS_CALL + bid * 64 + wid * 4), 1u, __ATOMIC_RELAXED, __HIP_MEMORY_SCOPE_AGENT) + 1;
    call = __builtin_amdgcn_readfirstlane(call);
    STAMP(0);
    const unsigned t0 = (unsigned)__builtin_amdgcn_s_memrealtime();
    if (tid == 0) ctl[0] = 0;
    WAIT_L0();
    BAR();
    for (int pc = gw; pc < NPC; pc += NCOMM * 8) {
#pragma unroll
      for (int i = 0; i < 4; ++i) v[i] = vn[i];
#pragma unroll
      for (int i = 0; i < 4; ++i) vn[i] = __builtin_amdgcn_raw_buffer_load_b128(sr, lane * 16 + i * 1024, (pc + NCOMM * 8) * 4096, 0);
      for (int e = 0; e < 7; ++e) {
        const unsigned pp = (unsigned)(rank + 1 + (e + gw) % 7) & 7u;
        const rsrc_t dr = mk_rsrc(P.ws[pp] + WS_X + ((call & 1) * NR + rank) * XSLOT, XSLOT);
#pragma unroll
        for (int i = 0; i < 4; ++i) __builtin_amdgcn_raw_buffer_store_b128(v[i], dr, lane * 16 + i * 1024, pc * 4096, AUX_SYS);
      }
      if (bid == 0 && lane == 0) {
        constexpr int NPW = NPC / NCOMM, NQ = NPW * 2 / 3;
        if (__hip_atomic_fetch_add(ctl, 1u, __ATOMIC_RELAXED, __HIP_MEMORY_SCOPE_WORKGROUP) == NQ - 1) {
          const unsigned te = t0 + ((unsigned)__builtin_amdgcn_s_memrealtime() - t0) * NPW / NQ + TSHIFT;
          __hip_atomic_store((unsigned long long*)(myws + WS_TST), ((unsigned long long)call << 32) | te, __ATOMIC_RELAXED, __HIP_MEMORY_SCOPE_AGENT);
        }
      }
    }
    WAIT_V(0);
    BAR();
    if (wid != 0) return;
    if (lane < 7) __hip_atomic_fetch_add((unsigned*)(P.ws[(rank + 1 + lane) & 7] + WS_XCNT + rank * 128), 1u, __ATOMIC_RELAXED, __HIP_MEMORY_SCOPE_SYSTEM);
    STAMP(5);
    if (bid == 0) {
      if (!wait_ge((unsigned*)(myws + WS_OCNT + (lane & 7) * 128), (lane & 7) == rank, call * 32u, tmo, call)) {
        for (int i = 0; i < 20000; ++i) __builtin_amdgcn_s_sleep(127);
        if (lane == 0) *(unsigned short*)(P.out[rank] + (size_t)rank * MR * NS * 2) = 0x7fc0;
      }
      __builtin_amdgcn_fence(__ATOMIC_ACQUIRE, "");
      STAMP(9);
    }
    return;
  }

  const int g = bid & 7, idx = (bid >> 3) - 2;
  const bool local = idx >= 28;
  const int i1 = local ? 0 : (idx & 1), cl = idx >> 1;
  const int d1 = cl % 7;
  const unsigned p1 = local ? (unsigned)rank : ((unsigned)(rank + 1 + d1) & 7u);
  const int c1 = g * 2 + (local ? idx - 28 : cl / 7);
  const int mt2 = (g & 1) * 7 + idx % 7;
  const int c2 = local ? c1 : (g >> 1) * 4 + idx / 7;
  const unsigned q2 = (unsigned)(rank + 1 + (mt2 >> 1)) & 7u;
  const int i2 = mt2 & 1;
  const int own2 = 18 * 16 + mt2 * 16;

  unsigned call = 0;
  if (wid == 7) {
    const rsrc_t xr = mk_rsrc(P.x, MR * KD * 2);
    unsigned a = 0;
    for (int i = (bid - GEMM0) * 64 + lane; i < MR * KD * 2 / 128; i += 240 * 64)
      a ^= __builtin_amdgcn_raw_buffer_load_b32(xr, (i & 511) * (KD * 2) + (i >> 9) * 128, 0, 0);
    asm volatile("" ::"v"(a));
    return;
  }
  if (wid == 6) {
    if (lane == 0) call = __hip_atomic_fetch_add((unsigned*)(myws + WS_CALL + bid * 64), 1u, __ATOMIC_RELAXED, __HIP_MEMORY_SCOPE_AGENT) + 1;
    call = __builtin_amdgcn_readfirstlane(call);
    if (lane == 0) ctl[0] = call;
    WAIT_L0();
    BAR();
    if (local) return;
    constexpr int SPARK = NKT + 1, SLAST = 2 * NKT + 3;
    int s = 0;
    const int ord = (c1 & 1) * 16 + i1 * 8 + ((g + d1) & 7);
    const unsigned long long* pw = (const unsigned long long*)(myws + ((lane & 1) ? WS_TST : WS_XCNT + q2 * 128));
    bool xok = false, tok = false, pushed = false, ok = true;
    unsigned slot = 0;
    auto look = [&](int smax) {
      unsigned long long v;
      asm volatile("global_load_dwordx2 %0, %1, off sc0 sc1" : "=v"(v) : "v"(pw) : "memory");
      pace(s, smax);
      asm volatile("s_waitcnt vmcnt(0)" : "+v"(v)::"memory");
      xok = xok || __builtin_amdgcn_readlane((unsigned)v, 0) >= call * (unsigned)NCOMM;
      if (!tok && __builtin_amdgcn_readlane((unsigned)(v >> 32), 1) == call) { tok = true; slot = __builtin_amdgcn_readlane((unsigned)v, 1); }
    };
    auto push = [&]() {
      STAMP(6);
      const rsrc_t dr = mk_rsrc(P.out[p1] + ((size_t)(rank * MR + i1 * BM1) * NS + c1 * BN) * 2, (BM1 - 1) * NS * 2 + 256);
#pragma unroll
      for (int h = 0; h < 2; ++h) {
        const int r0 = h * (BM1 / 2) + (lane >> 4);
        u32x4 v[H1 * 2];
#pragma unroll
        for (int i = 0; i < H1 * 2; ++i) { const int row = r0 + i * 4; v[i] = *(LAS u32x4*)(lds + STG_OFF + row * 256 + (((lane & 15) ^ (row & 7)) << 4)); }
#pragma unroll
        for (int i = 0; i < H1 * 2; ++i) __builtin_amdgcn_raw_buffer_store_b128(v[i], dr, r0 * (NS * 2) + (lane & 15) * 16, i * 4 * NS * 2, AUX_SYS);
      }
      pushed = true;
    };
    while (s < SPARK - 8) { BAR(); ++s; }
    while (s < SPARK - 1 && !(xok && tok)) { look(SPARK - 1); if (s < SPARK - 1) { BAR(); ++s; } }
    while (s < SPARK) { BAR(); ++s; }
    {
      const unsigned long long t0 = __builtin_amdgcn_s_memrealtime();
      auto due = [&]() { const unsigned now = (unsigned)__builtin_amdgcn_s_memrealtime() - ord * SLOT; return (int)(now - slot) >= 0 && (int)(now - (unsigned)t0) >= 0; };
      while (!xok) {
        if (!pushed && tok && due()) push();
        look(SPARK);
        if (xok) break;
        if (__builtin_amdgcn_readfirstlane(__hip_atomic_load(tmo, __ATOMIC_RELAXED, __HIP_MEMORY_SCOPE_AGENT)) == call ||
            __builtin_amdgcn_s_memrealtime() - t0 > DEADLINE) {
          __hip_atomic_store(tmo, call, __ATOMIC_RELAXED, __HIP_MEMORY_SCOPE_AGENT);
          break;
        }
      }
      __builtin_amdgcn_fence(__ATOMIC_ACQUIRE, "");
      WAIT_V(0);
      STAMP(8);
      BAR(); ++s;
      while (!pushed) {
        if (!tok) {
          look(SLAST);
          if (!tok && s >= SLAST && __builtin_amdgcn_s_memrealtime() - t0 > DEADLINE) { tok = true; slot = (unsigned)t0; }
        }
        if (tok && due()) { push(); break; }
        if (s < SLAST) { BAR(); ++s; } else __builtin_amdgcn_s_sleep(1);
      }
    }
    pace(s, SLAST);
    WAIT_V(0);
    if (ok && lane == 0) __hip_atomic_fetch_add((unsigned*)(P.ws[p1] + WS_OCNT + rank * 128), 1u, __ATOMIC_RELAXED, __HIP_MEMORY_SCOPE_SYSTEM);
    STAMP(7);
    return;
  }
  if (wid >= 4) {
    const int lw = wid - 4;
    const char* b2 = P.w + (size_t)(rank * NS + c2 * BN) * 2;
    if (local) {
      load_tile<H1, H1, 0>(lds, P.x, P.x, b2, lw, lane, ctl, call);
      load_tile<HL, HL, 0>(lds, P.x + (size_t)BM1 * KD * 2, P.x, b2, lw, lane, nullptr, call);
    } else {
      load_tile<H1, H1, AUXW1>(lds, P.x + (size_t)i1 * BM1 * KD * 2, P.x, P.w + (size_t)(p1 * NS + c1 * BN) * 2, lw, lane, ctl, call);
      BAR();
      BAR();
      load_tile<H2, HB, 0>(lds, myws + WS_X + ((call & 1) * NR + q2) * XSLOT + (size_t)i2 * HB * 16 * KD * 2, P.x + (size_t)own2 * KD * 2, b2, lw, lane, nullptr, call);
    }
    return;
  }
  const int wr = wid >> 1, wc = wid & 1, fr = lane & 15, fq = lane >> 4;
  const int coff = (wc * 64 + (fq & 1) * 16 + (fq >> 1) * 8) * 2;
  char* const o = P.out[rank];
  {
    constexpr int MT = H1 / 2;
    f32x4 acc[MT][4];
    STAMP(0);
    mfma_tile<MT>(acc, lds, H1, wr * MT, wc, lane, ctl, tmo, call);
    STAMP(1);
    if (local) {
      char* dst = o + ((size_t)(rank * MR + wr * MT * 16 + fr) * NS + c1 * BN) * 2 + coff;
#pragma unroll
      for (int m = 0; m < MT; ++m)
#pragma unroll
        for (int u = 0; u < 2; ++u) *(u32x4*)(dst + (size_t)m * 16 * NS * 2 + u * 64) = pack8(acc[m][2 * u], acc[m][2 * u + 1]);
    } else {
#pragma unroll
      for (int m = 0; m < MT; ++m)
#pragma unroll
        for (int u = 0; u < 2; ++u)
          *(LAS u32x4*)(lds + STG_OFF + (wr * MT * 16 + m * 16 + fr) * 256 + ((((coff >> 4) + u * 4) ^ (fr & 7)) << 4)) = pack8(acc[m][2 * u], acc[m][2 * u + 1]);
      WAIT_L0();
      BAR();
      BAR();
    }
    STAMP(2);
  }
  auto round2 = [&](auto mtc, int H, int s0) {
    constexpr int MT = decltype(mtc)::value;
    f32x4 acc[MT][4];
    unsigned tw;
    mfma_tile<MT>(acc, lds, H, s0, wc, lane, nullptr, tmo, tw);
    STAMP(3);
    const bool bad = tw == call;
#pragma unroll
    for (int m = 0; m < MT; ++m) {
      const int s = s0 + m;
      const int row = local ? rank * MR + BM1 + s * 16 : (s < HB ? (int)q2 * MR + RA + (i2 * HB + s) * 16 : rank * MR + own2);
      char* dst = o + ((size_t)(row + fr) * NS + c2 * BN) * 2 + coff;
#pragma unroll
      for (int u = 0; u < 2; ++u) {
        u32x4 v = pack8(acc[m][2 * u], acc[m][2 * u + 1]);
        if (bad) v = u32x4{0x7fc07fc0u, 0x7fc07fc0u, 0x7fc07fc0u, 0x7fc07fc0u};
        *(u32x4*)(dst + u * 64) = v;
      }
    }
    STAMP(4);
  };
  constexpr int MTU = (HL + 1) / 2;
  static_assert(HL % 2 == 0 && H2 == HL - 1, "round-2 split");
  if (wr == 0 || local) round2(std::integral_constant<int, MTU>{}, local ? HL : H2, wr * MTU);
  else round2(std::integral_constant<int, MTU - 1>{}, H2, MTU);
}
}

extern "C" void kernel_launch(int rank, int nranks, ncclComm_t comm, hipStream_t stream,
                              void* const* d_in, const int* in_sizes, int n_in,
                              void* d_out, int out_size, const int* in_dtypes, int out_dtype,
                              void* const* d_out_all, void* const* d_ws_all, size_t ws_bytes) {
  if (nranks != NR || (unsigned)rank >= (unsigned)NR || n_in < 2 || ws_bytes < WS_NEED ||
      in_sizes[0] != MR * KD || in_sizes[1] != KD * ND || out_size != NR * MR * NS) {
    fprintf(stderr, "kernel_launch: unexpected problem shape\n");
    return;
  }
  Params P;
  P.x = (const char*)d_in[0]; P.w = (const char*)d_in[1]; P.rank = rank;
  for (int p = 0; p < NR; ++p) { P.out[p] = (char*)d_out_all[p]; P.ws[p] = (char*)d_ws_all[p]; }
  gemm_a2a<<<256, 512, 0, stream>>>(P);
}
```

```cpp
#include <hip/hip_runtime.h>
#include <rccl/rccl.h>
#include <cstdio>
#include <type_traits>

namespace {
typedef __bf16 bf16x2 __attribute__((ext_vector_type(2)));
typedef __bf16 bf16x4 __attribute__((ext_vector_type(4)));
typedef __bf16 bf16x8 __attribute__((ext_vector_type(8)));
typedef float f32x2 __attribute__((ext_vector_type(2)));
typedef float f32x4 __attribute__((ext_vector_type(4)));
typedef unsigned u32x4 __attribute__((ext_vector_type(4)));
typedef unsigned long long u64;
#define LAS __attribute__((address_space(3)))

constexpr int NR = 8, MR = 512, KD = 4096, ND = 16384, NS = 2048;
constexpr int BK = 64, NKT = KD / BK, BN = 128;
constexpr int H1 = 10, H1B = 8, HB = 6;
constexpr int BM1 = 16 * H1, RA = 16 * (H1 + H1B), RB = MR - RA;
constexpr int BBYTES = BK * BN * 2;
constexpr int STG_OFF = 4 * (HB * 2048 + BBYTES);
constexpr int CTL_OFF = STG_OFF + H1 * 4096;
constexpr int LDS_BYTES = 160 * 1024;
static_assert(3 * ((HB + 2) * 2048 + BBYTES) <= STG_OFF && 4 * (H1 * 2048 + BBYTES) <= CTL_OFF && CTL_OFF + 2048 <= LDS_BYTES, "lds");
static_assert(RB == (2 * HB + 2) * 16 && H1B == HB + 2, "split");
constexpr int NCOMM = 16;
#define SLOT 84
#define XTWAIT 800
#define PMARG 110
#define NSB 1
#define TSHIFT 0

constexpr size_t WS_CALL = 0;
constexpr size_t WS_XCNT = 32768;
constexpr size_t WS_OCNT = 36864;
constexpr size_t WS_TMO = 40960;
constexpr size_t WS_TST = 45056;
constexpr size_t WS_XEND = 45312;
constexpr size_t WS_TICK = 49152;
constexpr size_t WS_BASE = 53248;
constexpr size_t WS_DBG = 131072;
constexpr size_t WS_X = 1u << 20;
constexpr size_t XSLOT = (size_t)RB * KD * 2;
constexpr size_t WS_NEED = WS_X + 2 * NR * XSLOT;
constexpr u64 DEADLINE = 40000000ull;

struct Params {
  const char* x; const char* w;
  char* out[NR]; char* ws[NR];
  int rank;
};

#define WAIT_V(n) asm volatile("s_waitcnt vmcnt(%0)" ::"n"(n) : "memory")
#define WAIT_L0() __builtin_amdgcn_s_waitcnt(0xC07F)
#define CFENCE() asm volatile("" ::: "memory")
#define BAR() do { CFENCE(); __builtin_amdgcn_s_barrier(); CFENCE(); } while (0)
#define NOW() __builtin_amdgcn_s_memrealtime()
#define RFL(x) __builtin_amdgcn_readfirstlane(x)

__device__ __forceinline__ unsigned pk(float a, float b) { f32x2 v = {a, b}; return __builtin_bit_cast(unsigned, __builtin_convertvector(v, bf16x2)); }
typedef __amdgpu_buffer_rsrc_t rsrc_t;
__device__ __forceinline__ rsrc_t mk_rsrc(const void* base, unsigned bytes) {
  const u64 b = (u64)base;
  const unsigned lo = RFL((unsigned)b), hi = RFL((unsigned)(b >> 32));
  return __builtin_amdgcn_make_buffer_rsrc((void*)(((u64)hi << 32) | lo), 0, bytes, 0x00020000);
}
constexpr int AUX_SYS = 17;

#define PACE_LOOP                                     \
  "1:\n\t"                                            \
  "s_getreg_b32 %[t], hwreg(HW_REG_IB_STS)\n\t"       \
  "s_and_b32 %[t], %[t], 0xc0000f\n\t"                \
  "s_cbranch_scc0 3f\n\t"                             \
  "s_cmp_ge_i32 %[s], %[m]\n\t"                       \
  "s_cbranch_scc1 2f\n\t"                             \
  "s_barrier\n\t"                                     \
  "s_add_i32 %[s], %[s], 1\n\t"                       \
  "s_branch 1b\n\t"                                   \
  "2:\n\t"                                            \
  "s_sleep 1\n\t"                                     \
  "s_branch 1b\n\t"                                   \
  "3:\n\t"                                            \
  "s_waitcnt vmcnt(0)"
__device__ __forceinline__ void pace(int& s, int smax) {
  unsigned t0; s = RFL(s);
  asm volatile(PACE_LOOP : [s] "+s"(s), [t] "=&s"(t0) : [m] "s"(smax) : "memory", "scc");
}
__device__ __forceinline__ u64 poll_paced(const void* p, int& s, int smax) {
  unsigned t0; u64 v; s = RFL(s);
  asm volatile("global_load_dwordx2 %[v], %[p], off sc0 sc1\n\t" PACE_LOOP
               : [s] "+s"(s), [t] "=&s"(t0), [v] "=&v"(v) : [m] "s"(smax), [p] "v"(p) : "memory", "scc");
  return v;
}
__device__ __forceinline__ void atomics_paced(void* pa, unsigned& a, void* pk_, u64& k, int& s, int smax) {
  unsigned t0; u64 ex; s = RFL(s);
  asm volatile("s_mov_b64 %[ex], exec\n\t"
               "s_mov_b64 exec, 1\n\t"
               "global_atomic_add %[a], %[pa], %[a], off sc0\n\t"
               "global_atomic_umax_x2 %[k], %[pk], %[k], off sc0\n\t"
               "s_mov_b64 exec, %[ex]\n\t" PACE_LOOP
               : [s] "+s"(s), [t] "=&s"(t0), [ex] "=&s"(ex), [a] "+v"(a), [k] "+v"(k) : [m] "s"(smax), [pa] "v"(pa), [pk] "v"(pk_) : "memory", "scc");
}
#define STAMP(i) do { } while (0)

template <int MT> struct Frag { bf16x8 a[MT]; bf16x8 b[4]; };

template <int MT>
__device__ __forceinline__ void rd(Frag<MT>& f, LAS char* base, const int (&ab)[2], const int (&bb)[4], int ks) {
#pragma unroll
  for (int n = 0; n < 4; ++n) {
    bf16x4 lo = __builtin_amdgcn_ds_read_tr16_b64_v4bf16((LAS bf16x4*)(base + bb[n] + (ks * 32) * 256));
    bf16x4 hi = __builtin_amdgcn_ds_read_tr16_b64_v4bf16((LAS bf16x4*)(base + bb[n] + (ks * 32 + 4) * 256));
    f.b[n] = __builtin_shufflevector(lo, hi, 0, 1, 2, 3, 4, 5, 6, 7);
  }
#pragma unroll
  for (int m = 0; m < MT; ++m) f.a[m] = *(LAS bf16x8*)(base + ab[ks] + m * 2048);
}
template <int MT>
__device__ __forceinline__ void mm(f32x4 (&acc)[MT][4], const Frag<MT>& f) {
#pragma unroll
  for (int m = 0; m < MT; ++m)
#pragma unroll
    for (int n = 0; n < 4; ++n) acc[m][n] = __builtin_amdgcn_mfma_f32_16x16x32_bf16(f.b[n], f.a[m], acc[m][n], 0, 0, 0);
}
template <int NMF, int NRD>
__device__ __forceinline__ void sched() {
  constexpr int NI = NRD < NMF ? NRD : NMF;
  if constexpr (NRD > NMF) __builtin_amdgcn_sched_group_barrier(0x100, NRD - NMF, 0);
#pragma unroll
  for (int i = 0; i < NI; ++i) { __builtin_amdgcn_sched_group_barrier(0x8, 1, 0); __builtin_amdgcn_sched_group_barrier(0x100, 1, 0); }
  if constexpr (NMF > NI) __builtin_amdgcn_sched_group_barrier(0x8, NMF - NI, 0);
}
__device__ __forceinline__ u32x4 pack8(const f32x4& c0, const f32x4& c1) {
  unsigned x0 = pk(c0[0], c0[1]), x1 = pk(c0[2], c0[3]), y0 = pk(c1[0], c1[1]), y1 = pk(c1[2], c1[3]);
  auto s0 = __builtin_amdgcn_permlane16_swap(x0, y0, false, false);
  auto s1 = __builtin_amdgcn_permlane16_swap(x1, y1, false, false);
  return u32x4{s0[0], s1[0], s0[1], s1[1]};
}

template <int MT, int NST>
__device__ __forceinline__ void mfma_tile(f32x4 (&acc)[MT][4], LAS char* lds, int wr, int wc, int lane) {
  constexpr int H = 2 * MT, STB = H * 2048 + BBYTES;
  const int fr = lane & 15, fq = lane >> 4;
  int ab[2], bb[4];
#pragma unroll
  for (int ks = 0; ks < 2; ++ks) ab[ks] = (wr * MT * 16 + fr) * 128 + (((ks * 4 + fq) ^ ((fr >> 1) & 7)) << 4);
  {
    const int q = (lane & 15) >> 2, p = lane & 3, k = 8 * fq + q;
    const int sw = (k & 3) | (((k >> 3) & 1) << 2);
#pragma unroll
    for (int n = 0; n < 4; ++n) bb[n] = H * 2048 + k * 256 + (((wc * 4 + n) ^ sw) << 5) + p * 8;
  }
#pragma unroll
  for (int m = 0; m < MT; ++m)
#pragma unroll
    for (int n = 0; n < 4; ++n) acc[m][n] = f32x4{0.f, 0.f, 0.f, 0.f};
  Frag<MT> f0, f1;
  rd<MT>(f0, lds, ab, bb, 0);
  int so = 0;
  for (int t = 0; t < NKT - 1; ++t) {
    rd<MT>(f1, lds + so, ab, bb, 1);
    mm<MT>(acc, f0);
    sched<MT * 4, MT + 8>();
    WAIT_L0();
    BAR();
    so = (so + STB == NST * STB) ? 0 : so + STB;
    rd<MT>(f0, lds + so, ab, bb, 0);
    mm<MT>(acc, f1);
    sched<MT * 4, MT + 8>();
  }
  rd<MT>(f1, lds + so, ab, bb, 1);
  mm<MT>(acc, f0);
  sched<MT * 4, MT + 8>();
  WAIT_L0();
  BAR();
  mm<MT>(acc, f1);
}

template <int H, int HA, int NST>
__device__ __forceinline__ void load_tile(LAS char* lds, const char* a, const char* a2, const char* b, int lw, int lane) {
  constexpr int STB = H * 2048 + BBYTES, D = H + 8;
  const int arow = lw * 8 + (lane >> 3);
  const unsigned va = (unsigned)(arow * (KD * 2) + (((lane & 7) ^ ((arow >> 1) & 7)) << 4));
  unsigned vb[2];
#pragma unroll
  for (int i = 0; i < 2; ++i) {
    const int k = (lw + 2 * i) * 4 + (lane >> 4);
    const int sw = (k & 3) | (((k >> 3) & 1) << 2);
    vb[i] = (unsigned)(k * (ND * 2) + (((lane & 15) ^ (sw << 1)) << 4));
  }
  int so = 0;
  auto issue = [&](int t) {
    LAS char* dst = lds + so + lw * 1024;
    so = (so + STB == NST * STB) ? 0 : so + STB;
    const char* ga = a + t * 128;
    const char* ga2 = a2 + t * 128;
    const char* gb = b + (size_t)t * BK * ND * 2;
#pragma unroll
    for (int i = 0; i < 8; ++i) __builtin_amdgcn_global_load_lds((const unsigned*)(gb + (size_t)(i >> 1) * 16 * ND * 2 + vb[i & 1]), (LAS unsigned*)(dst + H * 2048 + i * 2048), 16, 0, 0);
#pragma unroll
    for (int i = 0; i < H; ++i) __builtin_amdgcn_global_load_lds((const unsigned*)((i < HA ? ga + (size_t)i * 16 * KD * 2 : ga2 + (size_t)(i - HA) * 16 * KD * 2) + va), (LAS unsigned*)(dst + i * 2048), 16, 0, 0);
  };
#pragma unroll
  for (int t = 0; t < NST; ++t) issue(t);
  WAIT_V((NST - 1) * D); BAR();
  for (int t = 0; t < NKT - NST; ++t) { WAIT_V((NST - 2) * D); BAR(); issue(t + NST); }
  if constexpr (NST == 4) { WAIT_V(2 * D); BAR(); }
  WAIT_V(D); BAR();
  WAIT_V(0); BAR();
  BAR();
}

struct Tile2 { int big, peer, c2, mt, own; };
__device__ __forceinline__ Tile2 tile2(int g, int tk) {
  Tile2 t;
  const int par = g & 1, j = tk >> 2;
  const int own0 = par ? H1 + 12 : H1;
  t.c2 = (g >> 1) * 4 + (tk & 3);
  t.big = j < 6; t.peer = j > 0;
  const unsigned tab = par ? 0xCA8DB970u : 0x64205310u;
  t.mt = (tab >> (4 * j)) & 15;
  t.own = j == 0 ? own0 : (j == 5 ? own0 + 8 : ((j == 4 && !par) ? own0 + 10 : -1));
  return t;
}

__global__ __launch_bounds__(512, 2) void gemm_a2a(Params P) {
  __shared__ __attribute__((aligned(1024))) char smem[LDS_BYTES];
  LAS char* lds = (LAS char*)smem;
  const int tid = threadIdx.x, lane = tid & 63;
  if (tid == 0) STAMP(0);
  const int wid = RFL(tid >> 6);
  const int rank = P.rank;
  char* const myws = P.ws[rank];
  unsigned* const tmo = (unsigned*)(myws + WS_TMO);
  volatile LAS unsigned* ctl = (volatile LAS unsigned*)(lds + CTL_OFF);
  const int bid = blockIdx.x;
  const bool comm = bid < NCOMM;
  const int g = bid & 7, idx = (bid >> 3) - 2;
  const bool local = idx >= 28;
  const bool ab = !comm && !local;
  const int i1 = ab ? (idx & 1) : 0, cl = idx >> 1;
  const int d1 = ab ? cl % 7 : 0;
  const unsigned p1 = ab ? ((unsigned)(rank + 1 + d1) & 7u) : (unsigned)rank;
  const int c1 = ab ? ((g + d1) & 7) * 2 + cl / 7 : g * 2 + (local ? idx - 28 : 0);
  unsigned call = 0;

  if (comm) {
    constexpr int NPC = (int)(XSLOT / 2048), NW = NCOMM * 8, NPA = H1B * 16 * 4 / NW, NPT = NPC / NW;
    static_assert(NPC % NW == 0 && (H1B * 16 * 4) % NW == 0, "x pieces");
    const rsrc_t sr = mk_rsrc(P.x + (size_t)RA * KD * 2, XSLOT);
    const int gw = bid * 8 + wid;
    auto poff = [&](int j) { const int pc = j * NW + gw + HB * 16 * 4; return (pc >= NPC ? pc - NPC : pc) * 2048; };
    u32x4 v[NPT][2];
#pragma unroll
    for (int i = 0; i < 2; ++i) v[0][i] = __builtin_amdgcn_raw_buffer_load_b128(sr, lane * 16 + i * 1024, poff(0), 0);
    typedef __attribute__((address_space(1))) unsigned gu32;
    gu32* xc = (gu32*)(P.ws[(rank + 1 + (lane < 7 ? lane : 0)) & 7] + WS_XCNT + rank * 128);
    if (lane == 0) call = __hip_atomic_fetch_add((unsigned*)(myws + WS_CALL + bid * 64 + wid * 4), 1u, __ATOMIC_RELAXED, __HIP_MEMORY_SCOPE_AGENT) + 1;
    call = RFL(call);
    asm volatile("" : "+v"(xc));
#pragma unroll
    for (int j = 1; j < NPT; ++j)
#pragma unroll
      for (int i = 0; i < 2; ++i) v[j][i] = __builtin_amdgcn_raw_buffer_load_b128(sr, lane * 16 + i * 1024, poff(j), 0);
    STAMP(14);
    const unsigned t0 = (unsigned)NOW();
    if (tid == 0) { ctl[0] = call; ctl[1] = 0; ctl[2] = 0; WAIT_L0(); }
    if (bid == 0 && tid == 0) __hip_atomic_store((u64*)(myws + WS_TST), ((u64)call << 32) | t0, __ATOMIC_RELAXED, __HIP_MEMORY_SCOPE_AGENT);
#pragma unroll
    for (int j = 0; j < NPT; ++j) {
      if (j == NPT - 1 && tid == 0) {
        const unsigned now = (unsigned)NOW();
        __hip_atomic_store((u64*)(myws + WS_XEND) + bid, ((u64)call << 32) | (now + (now - t0) / (NPT - 1) + TSHIFT), __ATOMIC_RELAXED, __HIP_MEMORY_SCOPE_AGENT);
      }
      for (int e = 0; e < 7; ++e) {
        const unsigned pp = (unsigned)(rank + 1 + (e + gw) % 7) & 7u;
        const rsrc_t dr = mk_rsrc(P.ws[pp] + WS_X + ((call & 1) * NR + rank) * XSLOT, XSLOT);
#pragma unroll
        for (int i = 0; i < 2; ++i) __builtin_amdgcn_raw_buffer_store_b128(v[j][i], dr, lane * 16 + i * 1024, poff(j), AUX_SYS);
      }
      if (j == NPA) WAIT_V(14);
      if (j < NPT - 1) BAR();
      if (j == NPA && wid == 1 && lane < 7) __hip_atomic_fetch_add(xc + 16, 1u, __ATOMIC_RELAXED, __HIP_MEMORY_SCOPE_SYSTEM);
    }
    WAIT_V(0);
    BAR();
    if (wid == 7) {
      if (lane < 7) __hip_atomic_fetch_add(xc, 1u, __ATOMIC_RELAXED, __HIP_MEMORY_SCOPE_SYSTEM);
      STAMP(5);
    }
    if (wid == 7) return;
  }

  if (wid == 7) {
    for (int i0 = (bid - NCOMM) * 64; i0 < MR * KD * 2 / 128; i0 += 240 * 64) {
      const int i = i0 + lane;
      __builtin_amdgcn_global_load_lds((const unsigned*)(P.x + (size_t)(i & 511) * (KD * 2) + (i >> 9) * 128), (LAS unsigned*)(lds + CTL_OFF + 1024), 4, 0, 0);
    }
    return;
  }

  if (wid == 6) {
    constexpr int S1 = NKT, SLAST = 2 * NKT + 1;
    int s = S1;
    bool tok = !ab, pushed = !ab;
    constexpr int SLB = SLOT * H1B / H1;
    int soff = (c1 & 1) * 8 * (SLOT + SLB) + (i1 ? (c1 >> 1) * SLB : 8 * SLB + (c1 >> 1) * SLOT) / NSB;
    int part = 0;
    unsigned te = 0, d = 0;
    if (!comm) {
      if (lane == 0) call = __hip_atomic_fetch_add((unsigned*)(myws + WS_CALL + bid * 64), 1u, __ATOMIC_RELAXED, __HIP_MEMORY_SCOPE_AGENT) + 1;
      call = RFL(call);
      if (lane == 0) { ctl[0] = call; ctl[1] = 0; ctl[2] = 0; }
      WAIT_L0();
      BAR();
      s = 0;
      while (s < S1 - 20) { BAR(); ++s; }
      const unsigned ta = (unsigned)NOW();
      while (s < S1 - 10 && !tok) {
        const u64 v = poll_paced(myws + WS_TST, s, S1 - 10);
        if (RFL((unsigned)(v >> 32)) == call) { tok = true; te = RFL((unsigned)v); }
        else if (s < S1 - 10) { BAR(); ++s; }
      }
      while (s < S1 - 10) { BAR(); ++s; }
      if (ab && tok) {
        const unsigned tb = (unsigned)NOW();
        const int e = (int)(tb + (tb - ta) * 21 / 20 + PMARG - soff - te);
        d = e < 0 ? 0u : (unsigned)e;
      }
    }
    unsigned tk = 1u;
    u64 key = (ab && tok) ? (((u64)call << 32) | d) : 0ull;
    if (comm) tk = bid >> 3;
    else {
      atomics_paced(myws + WS_TICK + g * 128, tk, myws + WS_BASE + d1 * 128, key, s, S1);
      tk = RFL(tk) % 30u + 2u;
    }
    const Tile2 t2 = tile2(g, (int)tk);
    const unsigned q2 = (unsigned)(rank + 1 + (t2.mt >> 1)) & 7u;
    const char* pw = myws + (lane == 0 ? WS_XCNT + q2 * 128 + (t2.mt & 1) * 64 : lane == 1 ? WS_TST : lane == 2 ? WS_BASE + d1 * 128 : WS_XEND + (lane & 15) * 8);
    bool xok = !t2.peer, xend = false;
    unsigned tlook = (unsigned)NOW() - 100000u;
    auto look = [&](int smax) {
      tlook = (unsigned)NOW();
      const u64 v = poll_paced(pw, s, smax);
      xok = xok || __builtin_amdgcn_readlane((unsigned)v, 0) >= call * (unsigned)NCOMM;
      if (!tok && __builtin_amdgcn_readlane((unsigned)(v >> 32), 1) == call) { tok = true; te = __builtin_amdgcn_readlane((unsigned)v, 1); }
      if (__builtin_amdgcn_readlane((unsigned)(v >> 32), 2) == call) { const unsigned o = __builtin_amdgcn_readlane((unsigned)v, 2); d = o > d ? o : d; }
      if (!xend && tok && (unsigned)(__ballot((unsigned)(v >> 32) == call) >> 16 & 0xffffu) == 0xffffu) {
        xend = true;
        for (int i = 16; i < 32; ++i) { const unsigned o = __builtin_amdgcn_readlane((unsigned)v, i) - te; d = o > d ? o : d; }
      }
    };
    auto stale = [&]() { const unsigned slot = te + d + soff; return (int)((unsigned)NOW() + 150u - slot) >= 0 && part == 0 && (!xend || (int)(tlook + 250u - slot) < 0); };
    auto due = [&]() { return tok && (int)((unsigned)NOW() - (te + d + soff)) >= 0 && !stale() && RFL(ctl[2]) == 4u; };
    auto push = [&]() {
      CFENCE();
      if (part == 0) STAMP(6);
      const rsrc_t dr = mk_rsrc(P.out[p1] + ((size_t)(rank * MR + i1 * BM1) * NS + c1 * BN) * 2, ((i1 ? H1B : H1) * 16 - 1) * NS * 2 + 256);
      const int r0 = lane >> 4;
      const int o0 = STG_OFF + r0 * 256 + (((lane & 15) ^ r0) << 4), o1 = STG_OFF + (r0 + 4) * 256 + (((lane & 15) ^ (r0 + 4)) << 4);
      constexpr int NA = H1 * 4 / NSB, NB = H1B * 4 / NSB;
      static_assert(NA % 2 == 0 && NB % 2 == 0, "pieces start on a multiple of 8 rows");
      const int i0 = part * (i1 ? NB : NA);
      u32x4 v[NA];
#pragma unroll
      for (int i = 0; i < NA; ++i) v[i] = *(LAS u32x4*)(lds + ((i & 1) ? o1 : o0) + i0 * 1024 + (i >> 1) * 2048);
#pragma unroll
      for (int i = 0; i < NB; ++i) __builtin_amdgcn_raw_buffer_store_b128(v[i], dr, r0 * (NS * 2) + (lane & 15) * 16, (i0 + i) * 4 * NS * 2, AUX_SYS);
      if (!i1) {
#pragma unroll
        for (int i = NB; i < NA; ++i) __builtin_amdgcn_raw_buffer_store_b128(v[i], dr, r0 * (NS * 2) + (lane & 15) * 16, (i0 + i) * 4 * NS * 2, AUX_SYS);
      }
      soff += 8 * (SLOT + SLB) / NSB;
      pushed = ++part == NSB;
    };
    bool sent = !ab;
    auto signal = [&]() {
      if (lane == 0) __hip_atomic_fetch_add((unsigned*)(P.ws[p1] + WS_OCNT + rank * 128), 1u, __ATOMIC_RELAXED, __HIP_MEMORY_SCOPE_SYSTEM);
      STAMP(7);
      sent = true;
    };
    const u64 t0 = NOW();
    while (!xok) {
      if (!pushed && s >= S1 && due()) push();
      look(S1);
      if (pushed && !sent) signal();
      if (xok) break;
      if (s < S1) { BAR(); ++s; continue; }
      if (RFL(__hip_atomic_load(tmo, __ATOMIC_RELAXED, __HIP_MEMORY_SCOPE_AGENT)) == call || NOW() - t0 > DEADLINE) {
        __hip_atomic_store(tmo, call, __ATOMIC_RELAXED, __HIP_MEMORY_SCOPE_AGENT);
        break;
      }
    }
    if (t2.peer) { asm volatile("buffer_inv sc0 sc1" ::: "memory"); pace(s, S1); }
    if (lane == 0) ctl[1] = tk + 1;
    WAIT_L0();
    STAMP(8);
    while (!pushed) {
      const bool lk = !tok || stale();
      if (lk) look(SLAST);
      if ((s >= SLAST && NOW() - t0 > DEADLINE) || (s >= S1 && due())) { push(); continue; }
      if (s >= SLAST) __builtin_amdgcn_s_sleep(1);
      else if (!lk || !tok) { BAR(); ++s; }
    }
    if (!sent) { pace(s, SLAST); signal(); }
    if (bid == 0) {
      while (s < SLAST) { BAR(); ++s; }
      unsigned* of = (unsigned*)(myws + WS_OCNT + (lane & 7) * 128);
      const u64 t1 = NOW();
      for (;;) {
        const unsigned v = __hip_atomic_load(of, __ATOMIC_RELAXED, __HIP_MEMORY_SCOPE_SYSTEM);
        if (__all((lane & 7) == rank || v >= call * 32u)) break;
        if (RFL(__hip_atomic_load(tmo, __ATOMIC_RELAXED, __HIP_MEMORY_SCOPE_AGENT)) == call || NOW() - t1 > DEADLINE) {
          __hip_atomic_store(tmo, call, __ATOMIC_RELAXED, __HIP_MEMORY_SCOPE_AGENT);
          for (int i = 0; i < 20000; ++i) __builtin_amdgcn_s_sleep(127);
          if (lane == 0) *(unsigned short*)(P.out[rank] + (size_t)rank * MR * NS * 2) = 0x7fc0;
          break;
        }
        __builtin_amdgcn_s_sleep(2);
      }
      __builtin_amdgcn_fence(__ATOMIC_ACQUIRE, "");
      STAMP(9);
    }
    return;
  }

  if (wid >= 4) {
    const int lw = wid - 4;
    if (!comm) {
      const char* b1 = P.w + (size_t)(p1 * NS + c1 * BN) * 2;
      if (i1) load_tile<H1B, H1B, 4>(lds, P.x + (size_t)BM1 * KD * 2, P.x, b1, lw, lane);
      else load_tile<H1, H1, 4>(lds, P.x, P.x, b1, lw, lane);
    }
    unsigned tk;
    while ((tk = RFL(ctl[1])) == 0u) __builtin_amdgcn_s_sleep(1);
    call = RFL(ctl[0]);
    const Tile2 t2 = tile2(g, (int)tk - 1);
    const unsigned q2 = (unsigned)(rank + 1 + (t2.mt >> 1)) & 7u;
    const char* b2 = P.w + (size_t)(rank * NS + t2.c2 * BN) * 2;
    const char* own = P.x + (size_t)(t2.own < 0 ? 0 : t2.own) * 16 * KD * 2;
    const char* a = t2.peer ? myws + WS_X + ((call & 1) * NR + q2) * XSLOT + (size_t)(t2.mt & 1) * HB * 16 * KD * 2 : own;
    if (t2.big) load_tile<HB + 2, HB, 3>(lds, a, (t2.peer && t2.own >= 0) ? own : a + (size_t)HB * 16 * KD * 2, b2, lw, lane);
    else load_tile<HB, HB, 4>(lds, a, a, b2, lw, lane);
    return;
  }

  const int wr = wid >> 1, wc = wid & 1, fr = lane & 15, fq = lane >> 4;
  const int coff = (wc * 64 + (fq & 1) * 16 + (fq >> 1) * 8) * 2;
  char* const o = P.out[rank];
  auto round1 = [&](auto mtc) {
    constexpr int MT = decltype(mtc)::value;
    f32x4 acc[MT][4];
    BAR();
    STAMP(13);
    mfma_tile<MT, 4>(acc, lds, wr, wc, lane);
    STAMP(1);
    if (local) {
      char* dst = o + ((size_t)(rank * MR + wr * MT * 16 + fr) * NS + c1 * BN) * 2 + coff;
#pragma unroll
      for (int m = 0; m < MT; ++m)
#pragma unroll
        for (int u = 0; u < 2; ++u) *(u32x4*)(dst + (size_t)m * 16 * NS * 2 + u * 64) = pack8(acc[m][2 * u], acc[m][2 * u + 1]);
    } else {
#pragma unroll
      for (int m = 0; m < MT; ++m)
#pragma unroll
        for (int u = 0; u < 2; ++u)
          *(LAS u32x4*)(lds + STG_OFF + (wr * MT * 16 + m * 16 + fr) * 256 + ((((coff >> 4) + u * 4) ^ (fr & 7)) << 4)) = pack8(acc[m][2 * u], acc[m][2 * u + 1]);
      WAIT_L0();
      if (lane == 0) __hip_atomic_fetch_add((LAS unsigned*)(lds + CTL_OFF + 8), 1u, __ATOMIC_RELAXED, __HIP_MEMORY_SCOPE_WORKGROUP);
    }
    STAMP(2);
  };
  if (!comm) {
    if (i1) round1(std::integral_constant<int, H1B / 2>{});
    else round1(std::integral_constant<int, H1 / 2>{});
  }
  BAR();
  STAMP(10);
  call = RFL(ctl[0]);
  const Tile2 t2 = tile2(g, (int)RFL(ctl[1]) - 1);
  const unsigned tw = __hip_atomic_load(tmo, __ATOMIC_RELAXED, __HIP_MEMORY_SCOPE_AGENT);
  auto round2 = [&](auto mtc, auto nstc) {
    constexpr int MT = decltype(mtc)::value;
    f32x4 acc[MT][4];
    mfma_tile<MT, decltype(nstc)::value>(acc, lds, wr, wc, lane);
    STAMP(3);
    const bool bad = tw == call;
    const int q2 = (rank + 1 + (t2.mt >> 1)) & 7;
#pragma unroll
    for (int m = 0; m < MT; ++m) {
      const int s = wr * MT + m;
      const int row = !t2.peer ? rank * MR + (t2.own + s) * 16
                               : ((s < HB || t2.own < 0) ? q2 * MR + RA + ((t2.mt & 1) * HB + s) * 16 : rank * MR + (t2.own + s - HB) * 16);
      char* dst = o + ((size_t)(row + fr) * NS + t2.c2 * BN) * 2 + coff;
#pragma unroll
      for (int u = 0; u < 2; ++u) {
        u32x4 v = pack8(acc[m][2 * u], acc[m][2 * u + 1]);
        if (bad) v = u32x4{0x7fc07fc0u, 0x7fc07fc0u, 0x7fc07fc0u, 0x7fc07fc0u};
        *(u32x4*)(dst + u * 64) = v;
      }
    }
    STAMP(4);
  };
  if (t2.big) round2(std::integral_constant<int, HB / 2 + 1>{}, std::integral_constant<int, 3>{});
  else round2(std::integral_constant<int, HB / 2>{}, std::integral_constant<int, 4>{});
}
}

extern "C" void kernel_launch(int rank, int nranks, ncclComm_t comm, hipStream_t stream,
                              void* const* d_in, const int* in_sizes, int n_in,
                              void* d_out, int out_size, const int* in_dtypes, int out_dtype,
                              void* const* d_out_all, void* const* d_ws_all, size_t ws_bytes) {
  if (nranks != NR || (unsigned)rank >= (unsigned)NR || n_in < 2 || ws_bytes < WS_NEED ||
      in_sizes[0] != MR * KD || in_sizes[1] != KD * ND || out_size != NR * MR * NS) {
    fprintf(stderr, "kernel_launch: unexpected problem shape\n");
    return;
  }
  Params P;
  P.x = (const char*)d_in[0]; P.w = (const char*)d_in[1]; P.rank = rank;
  for (int p = 0; p < NR; ++p) { P.out[p] = (char*)d_out_all[p]; P.ws[p] = (char*)d_ws_all[p]; }
  gemm_a2a<<<256, 512, 0, stream>>>(P);
}
```

```cpp
#include <hip/hip_runtime.h>
#include <rccl/rccl.h>
#include <cstdio>
#include <type_traits>

namespace {
typedef __bf16 bf16x2 __attribute__((ext_vector_type(2)));
typedef __bf16 bf16x4 __attribute__((ext_vector_type(4)));
typedef __bf16 bf16x8 __attribute__((ext_vector_type(8)));
typedef float f32x2 __attribute__((ext_vector_type(2)));
typedef float f32x4 __attribute__((ext_vector_type(4)));
typedef unsigned u32x4 __attribute__((ext_vector_type(4)));
typedef unsigned long long u64;
#define LAS __attribute__((address_space(3)))

constexpr int NR = 8, MR = 512, KD = 4096, ND = 16384, NS = 2048;
constexpr int BK = 64, NKT = KD / BK, BN = 128;
constexpr int H1 = 10, H1B = 8, HB = 6;
constexpr int BM1 = 16 * H1, RA = 16 * (H1 + H1B), RB = MR - RA;
constexpr int BBYTES = BK * BN * 2;
constexpr int STG_OFF = 4 * (HB * 2048 + BBYTES);
constexpr int CTL_OFF = STG_OFF + H1 * 4096;
constexpr int LDS_BYTES = 160 * 1024;
static_assert(3 * ((HB + 2) * 2048 + BBYTES) <= STG_OFF && 4 * (H1 * 2048 + BBYTES) <= CTL_OFF && CTL_OFF + 2048 <= LDS_BYTES, "lds");
static_assert(RB == (2 * HB + 2) * 16 && H1B == HB + 2, "split");
constexpr int NCOMM = 16;
#define SLOT 84
#define PMARG 110
#define NSB 1
#define NST1 3
#define TSHIFT 0

constexpr size_t WS_CALL = 0;
constexpr size_t WS_XCNT = 32768;
constexpr size_t WS_OCNT = 36864;
constexpr size_t WS_TMO = 40960;
constexpr size_t WS_TST = 45056;
constexpr size_t WS_XEND = 45312;
constexpr size_t WS_TICK = 49152;
constexpr size_t WS_BASE = 53248;
constexpr size_t WS_DBG = 131072;
constexpr size_t WS_X = 1u << 20;
constexpr size_t XSLOT = (size_t)RB * KD * 2;
constexpr size_t WS_NEED = WS_X + 2 * NR * XSLOT;
constexpr u64 DEADLINE = 40000000ull;

struct Params {
  const char* x; const char* w;
  char* out[NR]; char* ws[NR];
  int rank;
};

#define WAIT_V(n) asm volatile("s_waitcnt vmcnt(%0)" ::"n"(n) : "memory")
#define WAIT_L0() __builtin_amdgcn_s_waitcnt(0xC07F)
#define CFENCE() asm volatile("" ::: "memory")
#define BAR() do { CFENCE(); __builtin_amdgcn_s_barrier(); CFENCE(); } while (0)
#define NOW() __builtin_amdgcn_s_memrealtime()
#define RFL(x) __builtin_amdgcn_readfirstlane(x)

__device__ __forceinline__ unsigned pk(float a, float b) { f32x2 v = {a, b}; return __builtin_bit_cast(unsigned, __builtin_convertvector(v, bf16x2)); }
typedef __amdgpu_buffer_rsrc_t rsrc_t;
__device__ __forceinline__ rsrc_t mk_rsrc(const void* base, unsigned bytes) {
  const u64 b = (u64)base;
  const unsigned lo = RFL((unsigned)b), hi = RFL((unsigned)(b >> 32));
  return __builtin_amdgcn_make_buffer_rsrc((void*)(((u64)hi << 32) | lo), 0, bytes, 0x00020000);
}
constexpr int AUX_SYS = 17;

#define PACE_LOOP                                     \
  "1:\n\t"                                            \
  "s_getreg_b32 %[t], hwreg(HW_REG_IB_STS)\n\t"       \
  "s_and_b32 %[t], %[t], 0xc0000f\n\t"                \
  "s_cbranch_scc0 3f\n\t"                             \
  "s_cmp_ge_i32 %[s], %[m]\n\t"                       \
  "s_cbranch_scc1 2f\n\t"                             \
  "s_barrier\n\t"                                     \
  "s_add_i32 %[s], %[s], 1\n\t"                       \
  "s_branch 1b\n\t"                                   \
  "2:\n\t"                                            \
  "s_sleep 1\n\t"                                     \
  "s_branch 1b\n\t"                                   \
  "3:\n\t"                                            \
  "s_waitcnt vmcnt(0)"
__device__ __forceinline__ void pace(int& s, int smax) {
  unsigned t0; s = RFL(s);
  asm volatile(PACE_LOOP : [s] "+s"(s), [t] "=&s"(t0) : [m] "s"(smax) : "memory", "scc");
}
__device__ __forceinline__ u64 poll_paced(const void* p, int& s, int smax) {
  unsigned t0; u64 v; s = RFL(s);
  asm volatile("global_load_dwordx2 %[v], %[p], off sc0 sc1\n\t" PACE_LOOP
               : [s] "+s"(s), [t] "=&s"(t0), [v] "=&v"(v) : [m] "s"(smax), [p] "v"(p) : "memory", "scc");
  return v;
}
__device__ __forceinline__ void atomics_paced(void* pa, unsigned& a, void* pk_, u64& k, int& s, int smax) {
  unsigned t0; u64 ex; s = RFL(s);
  asm volatile("s_mov_b64 %[ex], exec\n\t"
               "s_mov_b64 exec, 1\n\t"
               "global_atomic_add %[a], %[pa], %[a], off sc0\n\t"
               "global_atomic_umax_x2 %[k], %[pk], %[k], off sc0\n\t"
               "s_mov_b64 exec, %[ex]\n\t" PACE_LOOP
               : [s] "+s"(s), [t] "=&s"(t0), [ex] "=&s"(ex), [a] "+v"(a), [k] "+v"(k) : [m] "s"(smax), [pa] "v"(pa), [pk] "v"(pk_) : "memory", "scc");
}
#define STAMP(i) do { } while (0)
#define FLUSH() do { } while (0)

template <int MT> struct Frag { bf16x8 a[MT]; bf16x8 b[4]; };

template <int MT>
__device__ __forceinline__ void rd(Frag<MT>& f, LAS char* base, const int (&ab)[2], const int (&bb)[4], int ks) {
#pragma unroll
  for (int n = 0; n < 4; ++n) {
    bf16x4 lo = __builtin_amdgcn_ds_read_tr16_b64_v4bf16((LAS bf16x4*)(base + bb[n] + (ks * 32) * 256));
    bf16x4 hi = __builtin_amdgcn_ds_read_tr16_b64_v4bf16((LAS bf16x4*)(base + bb[n] + (ks * 32 + 4) * 256));
    f.b[n] = __builtin_shufflevector(lo, hi, 0, 1, 2, 3, 4, 5, 6, 7);
  }
#pragma unroll
  for (int m = 0; m < MT; ++m) f.a[m] = *(LAS bf16x8*)(base + ab[ks] + m * 2048);
}
template <int MT>
__device__ __forceinline__ void mm(f32x4 (&acc)[MT][4], const Frag<MT>& f) {
#pragma unroll
  for (int m = 0; m < MT; ++m)
#pragma unroll
    for (int n = 0; n < 4; ++n) acc[m][n] = __builtin_amdgcn_mfma_f32_16x16x32_bf16(f.b[n], f.a[m], acc[m][n], 0, 0, 0);
}
template <int NMF, int NRD>
__device__ __forceinline__ void sched() {
  constexpr int NI = NRD < NMF ? NRD : NMF;
  if constexpr (NRD > NMF) __builtin_amdgcn_sched_group_barrier(0x100, NRD - NMF, 0);
#pragma unroll
  for (int i = 0; i < NI; ++i) { __builtin_amdgcn_sched_group_barrier(0x8, 1, 0); __builtin_amdgcn_sched_group_barrier(0x100, 1, 0); }
  if constexpr (NMF > NI) __builtin_amdgcn_sched_group_barrier(0x8, NMF - NI, 0);
}
__device__ __forceinline__ u32x4 pack8(const f32x4& c0, const f32x4& c1) {
  unsigned x0 = pk(c0[0], c0[1]), x1 = pk(c0[2], c0[3]), y0 = pk(c1[0], c1[1]), y1 = pk(c1[2], c1[3]);
  auto s0 = __builtin_amdgcn_permlane16_swap(x0, y0, false, false);
  auto s1 = __builtin_amdgcn_permlane16_swap(x1, y1, false, false);
  return u32x4{s0[0], s1[0], s0[1], s1[1]};
}

template <int MT, int NST>
__device__ __forceinline__ void mfma_tile(f32x4 (&acc)[MT][4], LAS char* lds, int wr, int wc, int lane) {
  constexpr int H = 2 * MT, STB = H * 2048 + BBYTES;
  const int fr = lane & 15, fq = lane >> 4;
  int ab[2], bb[4];
#pragma unroll
  for (int ks = 0; ks < 2; ++ks) ab[ks] = (wr * MT * 16 + fr) * 128 + (((ks * 4 + fq) ^ ((fr >> 1) & 7)) << 4);
  {
    const int q = (lane & 15) >> 2, p = lane & 3, k = 8 * fq + q;
    const int sw = (k & 3) | (((k >> 3) & 1) << 2);
#pragma unroll
    for (int n = 0; n < 4; ++n) bb[n] = H * 2048 + k * 256 + (((wc * 4 + n) ^ sw) << 5) + p * 8;
  }
#pragma unroll
  for (int m = 0; m < MT; ++m)
#pragma unroll
    for (int n = 0; n < 4; ++n) acc[m][n] = f32x4{0.f, 0.f, 0.f, 0.f};
  Frag<MT> f0, f1;
  rd<MT>(f0, lds, ab, bb, 0);
  int so = 0;
  for (int t = 0; t < NKT - 1; ++t) {
    rd<MT>(f1, lds + so, ab, bb, 1);
    mm<MT>(acc, f0);
    sched<MT * 4, MT + 8>();
    WAIT_L0();
    BAR();
    so = (so + STB == NST * STB) ? 0 : so + STB;
    rd<MT>(f0, lds + so, ab, bb, 0);
    mm<MT>(acc, f1);
    sched<MT * 4, MT + 8>();
  }
  rd<MT>(f1, lds + so, ab, bb, 1);
  mm<MT>(acc, f0);
  sched<MT * 4, MT + 8>();
  WAIT_L0();
  BAR();
  mm<MT>(acc, f1);
}

template <int H, int HA, int NST>
__device__ __forceinline__ void load_tile(LAS char* lds, const char* a, const char* a2, const char* b, int lw, int lane) {
  constexpr int STB = H * 2048 + BBYTES, D = H + 8;
  const int arow = lw * 8 + (lane >> 3);
  const unsigned va = (unsigned)(arow * (KD * 2) + (((lane & 7) ^ ((arow >> 1) & 7)) << 4));
  unsigned vb[2];
#pragma unroll
  for (int i = 0; i < 2; ++i) {
    const int k = (lw + 2 * i) * 4 + (lane >> 4);
    const int sw = (k & 3) | (((k >> 3) & 1) << 2);
    vb[i] = (unsigned)(k * (ND * 2) + (((lane & 15) ^ (sw << 1)) << 4));
  }
  int so = 0;
  auto issue = [&](int t) {
    LAS char* dst = lds + so + lw * 1024;
    so = (so + STB == NST * STB) ? 0 : so + STB;
    const char* ga = a + t * 128;
    const char* ga2 = a2 + t * 128;
    const char* gb = b + (size_t)t * BK * ND * 2;
#pragma unroll
    for (int i = 0; i < 8; ++i) __builtin_amdgcn_global_load_lds((const unsigned*)(gb + (size_t)(i >> 1) * 16 * ND * 2 + vb[i & 1]), (LAS unsigned*)(dst + H * 2048 + i * 2048), 16, 0, 0);
#pragma unroll
    for (int i = 0; i < H; ++i) __builtin_amdgcn_global_load_lds((const unsigned*)((i < HA ? ga + (size_t)i * 16 * KD * 2 : ga2 + (size_t)(i - HA) * 16 * KD * 2) + va), (LAS unsigned*)(dst + i * 2048), 16, 0, 0);
  };
  issue(0); issue(1);
  WAIT_V(D); BAR();
#pragma unroll
  for (int t = 2; t < NST; ++t) issue(t);
  for (int t = 0; t < NKT - NST; ++t) { WAIT_V((NST - 2) * D); BAR(); issue(t + NST); }
  if constexpr (NST == 4) { WAIT_V(2 * D); BAR(); }
  WAIT_V(D); BAR();
  WAIT_V(0); BAR();
  BAR();
}

struct Tile2 { int big, peer, c2, mt, own; };
__device__ __forceinline__ Tile2 tile2(int g, int tk) {
  Tile2 t;
  const int par = g & 1, j = tk >> 2;
  t.c2 = (g >> 1) * 4 + ((tk & 3) ^ (par * 2));
  t.big = j < 6; t.peer = j > 0;
  const unsigned tab = par ? 0xCA8DB970u : 0x64205310u;
  t.mt = (tab >> (4 * j)) & 15;
  t.own = j == 0 ? 18 : (j == 5 ? 26 + par * 4 : ((j == 4 && !par) ? 28 : -1));
  return t;
}

__global__ __launch_bounds__(512, 2) void gemm_a2a(Params P) {
  __shared__ __attribute__((aligned(1024))) char smem[LDS_BYTES];
  LAS char* lds = (LAS char*)smem;
  const int tid = threadIdx.x, lane = tid & 63;
  if (tid == 0) STAMP(0);
  const int wid = RFL(tid >> 6);
  const int rank = P.rank;
  char* const myws = P.ws[rank];
  unsigned* const tmo = (unsigned*)(myws + WS_TMO);
  volatile LAS unsigned* ctl = (volatile LAS unsigned*)(lds + CTL_OFF);
  const int bid = blockIdx.x;
  const bool comm = bid < NCOMM;
  const int g = bid & 7, idx = (bid >> 3) - 2;
  const bool local = idx >= 28;
  const bool ab = !comm && !local;
  const int i1 = ab ? (idx & 1) : 0, cl = idx >> 1;
  const int d1 = ab ? cl % 7 : 0;
  const unsigned p1 = ab ? ((unsigned)(rank + 1 + d1) & 7u) : (unsigned)rank;
  const int c1 = ab ? ((g + d1) & 7) * 2 + cl / 7 : g * 2 + (local ? idx - 28 : bid >> 3);
  const int r1 = ab ? i1 * H1 : (comm ? H1B : 0);
  const bool small1 = i1 || local;
  unsigned call = 0;

  if (comm) {
    constexpr int NPC = (int)(XSLOT / 2048), NW = NCOMM * 8, NPA = H1B * 16 * 4 / NW, NPT = NPC / NW;
    static_assert(NPC % NW == 0 && (H1B * 16 * 4) % NW == 0, "x pieces");
    const rsrc_t sr = mk_rsrc(P.x + (size_t)RA * KD * 2, XSLOT);
    const int gw = bid * 8 + wid;
    auto poff = [&](int j) { const int pc = j * NW + gw + HB * 16 * 4; return (pc >= NPC ? pc - NPC : pc) * 2048; };
    u32x4 v[NPT][2];
#pragma unroll
    for (int i = 0; i < 2; ++i) v[0][i] = __builtin_amdgcn_raw_buffer_load_b128(sr, lane * 16 + i * 1024, poff(0), 0);
    typedef __attribute__((address_space(1))) unsigned gu32;
    gu32* xc = (gu32*)(P.ws[(rank + 1 + (lane < 7 ? lane : 0)) & 7] + WS_XCNT + rank * 128);
    if (lane == 0) call = __hip_atomic_fetch_add((unsigned*)(myws + WS_CALL + bid * 64 + wid * 4), 1u, __ATOMIC_RELAXED, __HIP_MEMORY_SCOPE_AGENT) + 1;
    call = RFL(call);
    asm volatile("" : "+v"(xc));
#pragma unroll
    for (int j = 1; j < NPT; ++j)
#pragma unroll
      for (int i = 0; i < 2; ++i) v[j][i] = __builtin_amdgcn_raw_buffer_load_b128(sr, lane * 16 + i * 1024, poff(j), 0);
    STAMP(14);
    const unsigned t0 = (unsigned)NOW();
    if (tid == 0) { ctl[0] = call; ctl[1] = 0; ctl[2] = 0; WAIT_L0(); }
    if (bid == 0 && tid == 0) __hip_atomic_store((u64*)(myws + WS_TST), ((u64)call << 32) | t0, __ATOMIC_RELAXED, __HIP_MEMORY_SCOPE_AGENT);
#pragma unroll
    for (int j = 0; j < NPT; ++j) {
      if (j == NPT - 1 && tid == 0) {
        const unsigned now = (unsigned)NOW();
        __hip_atomic_store((u64*)(myws + WS_XEND) + bid, ((u64)call << 32) | (now + (now - t0) / (NPT - 1) + TSHIFT), __ATOMIC_RELAXED, __HIP_MEMORY_SCOPE_AGENT);
      }
      for (int e = 0; e < 7; ++e) {
        const unsigned pp = (unsigned)(rank + 1 + (e + gw) % 7) & 7u;
        const rsrc_t dr = mk_rsrc(P.ws[pp] + WS_X + ((call & 1) * NR + rank) * XSLOT, XSLOT);
#pragma unroll
        for (int i = 0; i < 2; ++i) __builtin_amdgcn_raw_buffer_store_b128(v[j][i], dr, lane * 16 + i * 1024, poff(j), AUX_SYS);
      }
      if (j == NPA) WAIT_V(14);
      if (j < NPT - 1) BAR();
      if (j == NPA && wid == 1 && lane < 7) __hip_atomic_fetch_add(xc + 16, 1u, __ATOMIC_RELAXED, __HIP_MEMORY_SCOPE_SYSTEM);
    }
    WAIT_V(0);
    BAR();
    if (wid == 7) {
      if (lane < 7) __hip_atomic_fetch_add(xc, 1u, __ATOMIC_RELAXED, __HIP_MEMORY_SCOPE_SYSTEM);
      STAMP(5);
    }
    if (wid == 7) { FLUSH(); return; }
  }

  if (wid == 7) {
    for (int i0 = (bid - NCOMM) * 64; i0 < MR * KD * 2 / 128; i0 += 240 * 64) {
      const int i = i0 + lane;
      __builtin_amdgcn_global_load_lds((const unsigned*)(P.x + (size_t)(i & 511) * (KD * 2) + (i >> 9) * 128), (LAS unsigned*)(lds + CTL_OFF + 1024), 4, 0, 0);
    }
    return;
  }

  if (wid == 6) {
    constexpr int S1 = NKT, SLAST = 2 * NKT + 1;
    int s = S1;
    if (comm && bid != 0) { FLUSH(); return; }
    if (!comm) {
    bool tok = !ab, pushed = !ab;
    constexpr int SLB = SLOT * H1B / H1;
    int soff = (c1 & 1) * 8 * (SLOT + SLB) + (i1 ? (c1 >> 1) * SLB : 8 * SLB + (c1 >> 1) * SLOT) / NSB;
    int part = 0;
    unsigned te = 0, d = 0;
    if (!comm) {
      if (lane == 0) { ctl[1] = 0; ctl[2] = 0; }
      WAIT_L0();
      BAR();
      s = 0;
      {
        unsigned c = 1u; u64 k0 = 0;
        atomics_paced(myws + WS_CALL + bid * 64, c, myws + WS_CALL + bid * 64 + 8, k0, s, S1 - 20);
        call = RFL(c) + 1;
        if (lane == 0) ctl[0] = call;
      }
      while (s < S1 - 20) { BAR(); ++s; }
      const unsigned ta = (unsigned)NOW();
      while (s < S1 - 10 && !tok) {
        const u64 v = poll_paced(myws + WS_TST, s, S1 - 10);
        if (RFL((unsigned)(v >> 32)) == call) { tok = true; te = RFL((unsigned)v); }
        else if (s < S1 - 10) { BAR(); ++s; }
      }
      while (s < S1 - 10) { BAR(); ++s; }
      if (ab && tok) {
        const unsigned tb = (unsigned)NOW();
        const int e = (int)(tb + (tb - ta) * 21 / 20 + PMARG - soff - te);
        d = e < 0 ? 0u : (unsigned)e;
      }
    }
    unsigned tk = 1u;
    u64 key = (ab && tok) ? (((u64)call << 32) | d) : 0ull;
    atomics_paced(myws + WS_TICK + g * 128, tk, myws + WS_BASE + d1 * 128, key, s, S1);
    tk = RFL(tk) % 30u + 2u;
    const Tile2 t2 = tile2(g, (int)tk);
    const unsigned q2 = (unsigned)(rank + 1 + (t2.mt >> 1)) & 7u;
    const char* pw = myws + (lane == 0 ? WS_XCNT + q2 * 128 + (t2.mt & 1) * 64 : lane == 1 ? WS_TST : lane == 2 ? WS_BASE + d1 * 128 : WS_XEND + (lane & 15) * 8);
    bool xok = !t2.peer, xend = false;
    unsigned tlook = (unsigned)NOW() - 100000u;
    auto look = [&](int smax) {
      tlook = (unsigned)NOW();
      const u64 v = poll_paced(pw, s, smax);
      xok = xok || __builtin_amdgcn_readlane((unsigned)v, 0) >= call * (unsigned)NCOMM;
      if (!tok && __builtin_amdgcn_readlane((unsigned)(v >> 32), 1) == call) { tok = true; te = __builtin_amdgcn_readlane((unsigned)v, 1); }
      if (__builtin_amdgcn_readlane((unsigned)(v >> 32), 2) == call) { const unsigned o = __builtin_amdgcn_readlane((unsigned)v, 2); d = o > d ? o : d; }
      if (!xend && tok && (unsigned)(__ballot((unsigned)(v >> 32) == call) >> 16 & 0xffffu) == 0xffffu) {
        xend = true;
        for (int i = 16; i < 32; ++i) { const unsigned o = __builtin_amdgcn_readlane((unsigned)v, i) - te; d = o > d ? o : d; }
      }
    };
    auto stale = [&]() { const unsigned slot = te + d + soff; return (int)((unsigned)NOW() + 150u - slot) >= 0 && part == 0 && (!xend || (int)(tlook + 250u - slot) < 0); };
    auto due = [&]() { return tok && (int)((unsigned)NOW() - (te + d + soff)) >= 0 && !stale() && RFL(ctl[2]) == 4u; };
    auto push = [&]() {
      CFENCE();
      if (part == 0) STAMP(6);
      const rsrc_t dr = mk_rsrc(P.out[p1] + ((size_t)(rank * MR + i1 * BM1) * NS + c1 * BN) * 2, ((i1 ? H1B : H1) * 16 - 1) * NS * 2 + 256);
      const int r0 = lane >> 4;
      const int o0 = STG_OFF + r0 * 256 + (((lane & 15) ^ r0) << 4), o1 = STG_OFF + (r0 + 4) * 256 + (((lane & 15) ^ (r0 + 4)) << 4);
      constexpr int NA = H1 * 4 / NSB, NB = H1B * 4 / NSB;
      static_assert(NA % 2 == 0 && NB % 2 == 0, "pieces start on a multiple of 8 rows");
      const int i0 = part * (i1 ? NB : NA);
      u32x4 v[NA];
#pragma unroll
      for (int i = 0; i < NA; ++i) v[i] = *(LAS u32x4*)(lds + ((i & 1) ? o1 : o0) + i0 * 1024 + (i >> 1) * 2048);
#pragma unroll
      for (int i = 0; i < NB; ++i) __builtin_amdgcn_raw_buffer_store_b128(v[i], dr, r0 * (NS * 2) + (lane & 15) * 16, (i0 + i) * 4 * NS * 2, AUX_SYS);
      if (!i1) {
#pragma unroll
        for (int i = NB; i < NA; ++i) __builtin_amdgcn_raw_buffer_store_b128(v[i], dr, r0 * (NS * 2) + (lane & 15) * 16, (i0 + i) * 4 * NS * 2, AUX_SYS);
      }
      soff += 8 * (SLOT + SLB) / NSB;
      pushed = ++part == NSB;
    };
    bool sent = !ab;
    auto signal = [&]() {
      if (lane == 0) __hip_atomic_fetch_add((unsigned*)(P.ws[p1] + WS_OCNT + rank * 128), 1u, __ATOMIC_RELAXED, __HIP_MEMORY_SCOPE_SYSTEM);
      STAMP(7);
      sent = true;
    };
    const u64 t0 = NOW();
    while (!xok) {
      if (!pushed && s >= S1 && due()) push();
      look(S1);
      if (pushed && !sent) signal();
      if (xok) break;
      if (s < S1) { BAR(); ++s; continue; }
      if (RFL(__hip_atomic_load(tmo, __ATOMIC_RELAXED, __HIP_MEMORY_SCOPE_AGENT)) == call || NOW() - t0 > DEADLINE) {
        __hip_atomic_store(tmo, call, __ATOMIC_RELAXED, __HIP_MEMORY_SCOPE_AGENT);
        break;
      }
    }
    if (t2.peer) { asm volatile("buffer_inv sc0 sc1" ::: "memory"); pace(s, S1); }
    if (lane == 0) ctl[1] = tk + 1;
    WAIT_L0();
    STAMP(8);
    while (!pushed) {
      const bool lk = !tok || stale();
      if (lk) look(SLAST);
      if ((s >= SLAST && NOW() - t0 > DEADLINE) || (s >= S1 && due())) { push(); continue; }
      if (s >= SLAST) __builtin_amdgcn_s_sleep(1);
      else if (!lk || !tok) { BAR(); ++s; }
    }
    if (!sent) { pace(s, SLAST); signal(); }
    } else {
      while (s < SLAST) { BAR(); ++s; }
      unsigned* of = (unsigned*)(myws + WS_OCNT + (lane & 7) * 128);
      const u64 t1 = NOW();
      for (;;) {
        const unsigned v = __hip_atomic_load(of, __ATOMIC_RELAXED, __HIP_MEMORY_SCOPE_SYSTEM);
        if (__all((lane & 7) == rank || v >= call * 32u)) break;
        if (RFL(__hip_atomic_load(tmo, __ATOMIC_RELAXED, __HIP_MEMORY_SCOPE_AGENT)) == call || NOW() - t1 > DEADLINE) {
          __hip_atomic_store(tmo, call, __ATOMIC_RELAXED, __HIP_MEMORY_SCOPE_AGENT);
          for (int i = 0; i < 20000; ++i) __builtin_amdgcn_s_sleep(127);
          if (lane == 0) *(unsigned short*)(P.out[rank] + (size_t)rank * MR * NS * 2) = 0x7fc0;
          break;
        }
        __builtin_amdgcn_s_sleep(2);
      }
      __builtin_amdgcn_fence(__ATOMIC_ACQUIRE, "");
      STAMP(9);
    }
    FLUSH();
    return;
  }

  if (wid >= 4) {
    const int lw = wid - 4;
    {
      const char* b1 = P.w + (size_t)(p1 * NS + c1 * BN) * 2;
      const char* a1 = P.x + (size_t)r1 * 16 * KD * 2;
      if (small1) load_tile<H1B, H1B, NST1>(lds, a1, P.x, b1, lw, lane);
      else load_tile<H1, H1, NST1>(lds, a1, P.x, b1, lw, lane);
      if (comm) return;
    }
    unsigned tk;
    while ((tk = RFL(ctl[1])) == 0u) __builtin_amdgcn_s_sleep(1);
    call = RFL(ctl[0]);
    const Tile2 t2 = tile2(g, (int)tk - 1);
    const unsigned q2 = (unsigned)(rank + 1 + (t2.mt >> 1)) & 7u;
    const char* b2 = P.w + (size_t)(rank * NS + t2.c2 * BN) * 2;
    const char* own = P.x + (size_t)(t2.own < 0 ? 0 : t2.own) * 16 * KD * 2;
    const char* a = t2.peer ? myws + WS_X + ((call & 1) * NR + q2) * XSLOT + (size_t)(t2.mt & 1) * HB * 16 * KD * 2 : own;
    if (t2.big) load_tile<HB + 2, HB, 3>(lds, a, (t2.peer && t2.own >= 0) ? own : a + (size_t)HB * 16 * KD * 2, b2, lw, lane);
    else load_tile<HB, HB, 4>(lds, a, a, b2, lw, lane);
    return;
  }

  const int wr = wid >> 1, wc = wid & 1, fr = lane & 15, fq = lane >> 4;
  const int coff = (wc * 64 + (fq & 1) * 16 + (fq >> 1) * 8) * 2;
  char* const o = P.out[rank];
  auto round1 = [&](auto mtc) {
    constexpr int MT = decltype(mtc)::value;
    f32x4 acc[MT][4];
    BAR();
    STAMP(13);
    mfma_tile<MT, NST1>(acc, lds, wr, wc, lane);
    STAMP(1);
    if (!ab) {
      char* dst = o + ((size_t)(rank * MR + (r1 + wr * MT) * 16 + fr) * NS + c1 * BN) * 2 + coff;
#pragma unroll
      for (int m = 0; m < MT; ++m)
#pragma unroll
        for (int u = 0; u < 2; ++u) *(u32x4*)(dst + (size_t)m * 16 * NS * 2 + u * 64) = pack8(acc[m][2 * u], acc[m][2 * u + 1]);
    } else {
#pragma unroll
      for (int m = 0; m < MT; ++m)
#pragma unroll
        for (int u = 0; u < 2; ++u)
          *(LAS u32x4*)(lds + STG_OFF + (wr * MT * 16 + m * 16 + fr) * 256 + ((((coff >> 4) + u * 4) ^ (fr & 7)) << 4)) = pack8(acc[m][2 * u], acc[m][2 * u + 1]);
      WAIT_L0();
      if (lane == 0) __hip_atomic_fetch_add((LAS unsigned*)(lds + CTL_OFF + 8), 1u, __ATOMIC_RELAXED, __HIP_MEMORY_SCOPE_WORKGROUP);
    }
    STAMP(2);
  };
  if (small1) round1(std::integral_constant<int, H1B / 2>{});
  else round1(std::integral_constant<int, H1 / 2>{});
  if (comm) { if (wid == 0) FLUSH(); return; }
  BAR();
  STAMP(10);
  call = RFL(ctl[0]);
  const Tile2 t2 = tile2(g, (int)RFL(ctl[1]) - 1);
  const unsigned tw = __hip_atomic_load(tmo, __ATOMIC_RELAXED, __HIP_MEMORY_SCOPE_AGENT);
  auto round2 = [&](auto mtc, auto nstc) {
    constexpr int MT = decltype(mtc)::value;
    f32x4 acc[MT][4];
    mfma_tile<MT, decltype(nstc)::value>(acc, lds, wr, wc, lane);
    STAMP(3);
    const bool bad = tw == call;
    const int q2 = (rank + 1 + (t2.mt >> 1)) & 7;
#pragma unroll
    for (int m = 0; m < MT; ++m) {
      const int s = wr * MT + m;
      const int row = !t2.peer ? rank * MR + (t2.own + s) * 16
                               : ((s < HB || t2.own < 0) ? q2 * MR + RA + ((t2.mt & 1) * HB + s) * 16 : rank * MR + (t2.own + s - HB) * 16);
      char* dst = o + ((size_t)(row + fr) * NS + t2.c2 * BN) * 2 + coff;
#pragma unroll
      for (int u = 0; u < 2; ++u) {
        u32x4 v = pack8(acc[m][2 * u], acc[m][2 * u + 1]);
        if (bad) v = u32x4{0x7fc07fc0u, 0x7fc07fc0u, 0x7fc07fc0u, 0x7fc07fc0u};
        asm volatile("global_store_dwordx4 %0, %1, off sc1\n\ts_nop 1" ::"v"(dst + u * 64), "v"(v) : "memory");
      }
    }
    STAMP(4);
    if (wid == 0) FLUSH();
  };
  if (t2.big) round2(std::integral_constant<int, HB / 2 + 1>{}, std::integral_constant<int, 3>{});
  else round2(std::integral_constant<int, HB / 2>{}, std::integral_constant<int, 4>{});
}
}

extern "C" void kernel_launch(int rank, int nranks, ncclComm_t comm, hipStream_t stream,
                              void* const* d_in, const int* in_sizes, int n_in,
                              void* d_out, int out_size, const int* in_dtypes, int out_dtype,
                              void* const* d_out_all, void* const* d_ws_all, size_t ws_bytes) {
  if (nranks != NR || (unsigned)rank >= (unsigned)NR || n_in < 2 || ws_bytes < WS_NEED ||
      in_sizes[0] != MR * KD || in_sizes[1] != KD * ND || out_size != NR * MR * NS) {
    fprintf(stderr, "kernel_launch: unexpected problem shape\n");
    return;
  }
  Params P;
  P.x = (const char*)d_in[0]; P.w = (const char*)d_in[1]; P.rank = rank;
  for (int p = 0; p < NR; ++p) { P.out[p] = (char*)d_out_all[p]; P.ws[p] = (char*)d_ws_all[p]; }
  gemm_a2a<<<256, 512, 0, stream>>>(P);
}
```

```cpp
#include <hip/hip_runtime.h>
#include <rccl/rccl.h>
#include <cstdio>
#include <type_traits>

namespace {
typedef __bf16 bf16x2 __attribute__((ext_vector_type(2)));
typedef __bf16 bf16x4 __attribute__((ext_vector_type(4)));
typedef __bf16 bf16x8 __attribute__((ext_vector_type(8)));
typedef float f32x2 __attribute__((ext_vector_type(2)));
typedef float f32x4 __attribute__((ext_vector_type(4)));
typedef unsigned u32x4 __attribute__((ext_vector_type(4)));
typedef unsigned long long u64;
#define LAS __attribute__((address_space(3)))

constexpr int NR = 8, MR = 512, KD = 4096, ND = 16384, NS = 2048;
constexpr int BK = 64, NKT = KD / BK, BN = 128;
constexpr int H1 = 10, H1B = 8, HB = 6;
constexpr int BM1 = 16 * H1, RA = 16 * (H1 + H1B), RB = MR - RA;
constexpr int BBYTES = BK * BN * 2;
constexpr int STG_OFF = 4 * (HB * 2048 + BBYTES);
constexpr int CTL_OFF = STG_OFF + H1 * 4096;
constexpr int LDS_BYTES = 160 * 1024;
static_assert(3 * ((HB + 2) * 2048 + BBYTES) <= STG_OFF && 4 * (H1 * 2048 + BBYTES) <= CTL_OFF && CTL_OFF + 2048 <= LDS_BYTES, "lds");
static_assert(RB == (2 * HB + 2) * 16 && H1B == HB + 2, "split");
constexpr int NCOMM = 16;
constexpr int SLOT = 84;
constexpr int PMARG = 110;
constexpr int NST1 = 3;

constexpr size_t WS_CALL = 0;
constexpr size_t WS_XCNT = 32768;
constexpr size_t WS_OCNT = 36864;
constexpr size_t WS_TMO = 40960;
constexpr size_t WS_TST = 45056;
constexpr size_t WS_XEND = 45312;
constexpr size_t WS_TICK = 49152;
constexpr size_t WS_BASE = 53248;
constexpr size_t WS_X = 1u << 20;
constexpr size_t XSLOT = (size_t)RB * KD * 2;
constexpr size_t WS_NEED = WS_X + 2 * NR * XSLOT;
constexpr u64 DEADLINE = 80000000ull;

struct Params {
  const char* x; const char* w;
  char* out[NR]; char* ws[NR];
  int rank;
};

#define WAIT_V(n) asm volatile("s_waitcnt vmcnt(%0)" ::"n"(n) : "memory")
#define WAIT_L0() __builtin_amdgcn_s_waitcnt(0xC07F)
#define CFENCE() asm volatile("" ::: "memory")
#define BAR() do { CFENCE(); __builtin_amdgcn_s_barrier(); CFENCE(); } while (0)
#define NOW() __builtin_amdgcn_s_memrealtime()
#define RFL(x) __builtin_amdgcn_readfirstlane(x)

__device__ __forceinline__ unsigned pk(float a, float b) { f32x2 v = {a, b}; return __builtin_bit_cast(unsigned, __builtin_convertvector(v, bf16x2)); }
typedef __amdgpu_buffer_rsrc_t rsrc_t;
__device__ __forceinline__ rsrc_t mk_rsrc(const void* base, unsigned bytes) {
  const u64 b = (u64)base;
  const unsigned lo = RFL((unsigned)b), hi = RFL((unsigned)(b >> 32));
  return __builtin_amdgcn_make_buffer_rsrc((void*)(((u64)hi << 32) | lo), 0, bytes, 0x00020000);
}
constexpr int AUX_SYS = 17;

#define PACE_LOOP                                     \
  "1:\n\t"                                            \
  "s_getreg_b32 %[t], hwreg(HW_REG_IB_STS)\n\t"       \
  "s_and_b32 %[t], %[t], 0xc0000f\n\t"                \
  "s_cbranch_scc0 3f\n\t"                             \
  "s_cmp_ge_i32 %[s], %[m]\n\t"                       \
  "s_cbranch_scc1 2f\n\t"                             \
  "s_barrier\n\t"                                     \
  "s_add_i32 %[s], %[s], 1\n\t"                       \
  "s_branch 1b\n\t"                                   \
  "2:\n\t"                                            \
  "s_sleep 1\n\t"                                     \
  "s_branch 1b\n\t"                                   \
  "3:\n\t"                                            \
  "s_waitcnt vmcnt(0)"
__device__ __forceinline__ void pace(int& s, int smax) {
  unsigned t0; s = RFL(s);
  asm volatile(PACE_LOOP : [s] "+s"(s), [t] "=&s"(t0) : [m] "s"(smax) : "memory", "scc");
}
__device__ __forceinline__ u64 poll_paced(const void* p, int& s, int smax) {
  unsigned t0; u64 v; s = RFL(s);
  asm volatile("global_load_dwordx2 %[v], %[p], off sc0 sc1\n\t" PACE_LOOP
               : [s] "+s"(s), [t] "=&s"(t0), [v] "=&v"(v) : [m] "s"(smax), [p] "v"(p) : "memory", "scc");
  return v;
}
__device__ __forceinline__ void atomics_paced(void* pa, unsigned& a, void* pk_, u64& k, int& s, int smax) {
  unsigned t0; u64 ex; s = RFL(s);
  asm volatile("s_mov_b64 %[ex], exec\n\t"
               "s_mov_b64 exec, 1\n\t"
               "global_atomic_add %[a], %[pa], %[a], off sc0\n\t"
               "global_atomic_umax_x2 %[k], %[pk], %[k], off sc0\n\t"
               "s_mov_b64 exec, %[ex]\n\t" PACE_LOOP
               : [s] "+s"(s), [t] "=&s"(t0), [ex] "=&s"(ex), [a] "+v"(a), [k] "+v"(k) : [m] "s"(smax), [pa] "v"(pa), [pk] "v"(pk_) : "memory", "scc");
}

template <int MT> struct Frag { bf16x8 a[MT]; bf16x8 b[4]; };

template <int MT>
__device__ __forceinline__ void rd(Frag<MT>& f, LAS char* base, const int (&ab)[2], const int (&bb)[4], int ks) {
  f.a[0] = *(LAS bf16x8*)(base + ab[ks]);
#pragma unroll
  for (int n = 0; n < 4; ++n) {
    bf16x4 lo = __builtin_amdgcn_ds_read_tr16_b64_v4bf16((LAS bf16x4*)(base + bb[n] + (ks * 32) * 256));
    bf16x4 hi = __builtin_amdgcn_ds_read_tr16_b64_v4bf16((LAS bf16x4*)(base + bb[n] + (ks * 32 + 4) * 256));
    f.b[n] = __builtin_shufflevector(lo, hi, 0, 1, 2, 3, 4, 5, 6, 7);
  }
#pragma unroll
  for (int m = 1; m < MT; ++m) f.a[m] = *(LAS bf16x8*)(base + ab[ks] + m * 2048);
}
template <int MT, int M0, int M1>
__device__ __forceinline__ void mm(f32x4 (&acc)[MT][4], const Frag<MT>& f) {
#pragma unroll
  for (int m = M0; m < M1; ++m)
#pragma unroll
    for (int j = 0; j < 4; ++j) { const int n = (m & 1) ? 3 - j : j; acc[m][n] = __builtin_amdgcn_mfma_f32_16x16x32_bf16(f.b[n], f.a[m], acc[m][n], 0, 0, 0); }
}
template <int NMF, int NRD>
__device__ __forceinline__ void sched() {
  constexpr int NI = NRD < NMF ? NRD : NMF, DBL = NRD - NI;
#pragma unroll
  for (int i = 0; i < NI; ++i) {
    __builtin_amdgcn_sched_group_barrier(0x8, 1, 0); __builtin_amdgcn_sched_group_barrier(0x100, 1, 0);
    if (i < DBL) __builtin_amdgcn_sched_group_barrier(0x100, 1, 0);
  }
  if constexpr (NMF > NI) __builtin_amdgcn_sched_group_barrier(0x8, NMF - NI, 0);
}
__device__ __forceinline__ u32x4 pack8(const f32x4& c0, const f32x4& c1) {
  unsigned x0 = pk(c0[0], c0[1]), x1 = pk(c0[2], c0[3]), y0 = pk(c1[0], c1[1]), y1 = pk(c1[2], c1[3]);
  auto s0 = __builtin_amdgcn_permlane16_swap(x0, y0, false, false);
  auto s1 = __builtin_amdgcn_permlane16_swap(x1, y1, false, false);
  return u32x4{s0[0], s1[0], s0[1], s1[1]};
}

template <int MT, int NST>
__device__ __forceinline__ void mfma_tile(f32x4 (&acc)[MT][4], LAS char* lds, int wr, int wc, int lane) {
  constexpr int H = 2 * MT, STB = H * 2048 + BBYTES;
  const int fr = lane & 15, fq = lane >> 4;
  int ab[2], bb[4];
#pragma unroll
  for (int ks = 0; ks < 2; ++ks) ab[ks] = (wr * MT * 16 + fr) * 128 + (((ks * 4 + fq) ^ ((fr >> 1) & 7)) << 4);
  {
    const int q = (lane & 15) >> 2, p = lane & 3, k = 8 * fq + q;
    const int sw = (k & 3) | (((k >> 3) & 1) << 2);
#pragma unroll
    for (int n = 0; n < 4; ++n) bb[n] = H * 2048 + k * 256 + (((wc * 4 + n) ^ sw) << 5) + p * 8;
  }
#pragma unroll
  for (int m = 0; m < MT; ++m)
#pragma unroll
    for (int n = 0; n < 4; ++n) acc[m][n] = f32x4{0.f, 0.f, 0.f, 0.f};
  Frag<MT> f0, f1;
  rd<MT>(f0, lds, ab, bb, 0);
  WAIT_L0();
  int so = 0;
  for (int t = 0; t < NKT - 1; ++t) {
    rd<MT>(f1, lds + so, ab, bb, 1);
    mm<MT, 0, MT>(acc, f0);
    sched<MT * 4, MT + 8>();
    __builtin_amdgcn_sched_barrier(0);
    mm<MT, 0, 1>(acc, f1);
    __builtin_amdgcn_sched_barrier(0);
    WAIT_L0();
    BAR();
    so = (so + STB == NST * STB) ? 0 : so + STB;
    rd<MT>(f0, lds + so, ab, bb, 0);
    mm<MT, 1, MT>(acc, f1);
    sched<MT * 4 - 4, MT + 8>();
  }
  rd<MT>(f1, lds + so, ab, bb, 1);
  mm<MT, 0, MT>(acc, f0);
  sched<MT * 4, MT + 8>();
  __builtin_amdgcn_sched_barrier(0);
  WAIT_L0();
  BAR();
  mm<MT, 0, MT>(acc, f1);
}

template <int H, int HA, int NST>
__device__ __forceinline__ void load_tile(LAS char* lds, const char* a, const char* a2, const char* b, int lw, int lane) {
  constexpr int STB = H * 2048 + BBYTES, D = H + 8;
  const int arow = lw * 8 + (lane >> 3);
  const unsigned va = (unsigned)(arow * (KD * 2) + (((lane & 7) ^ ((arow >> 1) & 7)) << 4));
  unsigned vb[2];
#pragma unroll
  for (int i = 0; i < 2; ++i) {
    const int k = (lw + 2 * i) * 4 + (lane >> 4);
    const int sw = (k & 3) | (((k >> 3) & 1) << 2);
    vb[i] = (unsigned)(k * (ND * 2) + (((lane & 15) ^ (sw << 1)) << 4));
  }
  int so = 0;
  const rsrc_t ra = mk_rsrc(a, HA * 16 * KD * 2), ra2 = mk_rsrc(a2, (H - HA) * 16 * KD * 2), rb = mk_rsrc(b, (unsigned)((KD - 1) * ND * 2 + 256));
  auto issue = [&](int t) {
    LAS char* dst = lds + so + lw * 1024;
    so = (so + STB == NST * STB) ? 0 : so + STB;
#pragma unroll
    for (int i = 0; i < 8; ++i)
      __builtin_amdgcn_raw_ptr_buffer_load_lds(rb, dst + H * 2048 + i * 2048, 16, vb[i & 1], (t * BK + (i >> 1) * 16) * (ND * 2), 0, 0);
#pragma unroll
    for (int i = 0; i < H; ++i)
      __builtin_amdgcn_raw_ptr_buffer_load_lds(i < HA ? ra : ra2, dst + i * 2048, 16, va, t * 128 + (i < HA ? i : i - HA) * 16 * KD * 2, 0, 0);
  };
  issue(0); issue(1);
  WAIT_V(D); BAR();
#pragma unroll
  for (int t = 2; t < NST; ++t) issue(t);
  for (int t = 0; t < NKT - NST; ++t) { WAIT_V((NST - 2) * D); BAR(); issue(t + NST); }
  if constexpr (NST == 4) { WAIT_V(2 * D); BAR(); }
  WAIT_V(D); BAR();
  WAIT_V(0); BAR();
  BAR();
}

struct Tile2 { int big, peer, c2, mt, own; };
__device__ __forceinline__ Tile2 tile2(int g, int tk) {
  Tile2 t;
  const int par = g & 1, j = tk >> 2;
  t.c2 = (g >> 1) * 4 + ((tk & 3) ^ (par * 2));
  t.big = j < 6; t.peer = j > 0;
  const unsigned tab = par ? 0xCA8DB970u : 0x64205310u;
  t.mt = (tab >> (4 * j)) & 15;
  t.own = j == 0 ? 18 : (j == 5 ? 26 + par * 4 : ((j == 4 && !par) ? 28 : -1));
  return t;
}

__global__ __launch_bounds__(512, 2) void gemm_a2a(Params P) {
  __shared__ __attribute__((aligned(1024))) char smem[LDS_BYTES];
  LAS char* lds = (LAS char*)smem;
  const int tid = threadIdx.x, lane = tid & 63;
  const int wid = RFL(tid >> 6);
  const int rank = P.rank;
  char* const myws = P.ws[rank];
  unsigned* const tmo = (unsigned*)(myws + WS_TMO);
  volatile LAS unsigned* ctl = (volatile LAS unsigned*)(lds + CTL_OFF);
  const int bid = blockIdx.x;
  const bool comm = bid < NCOMM;
  const int g = bid & 7, idx = (bid >> 3) - 2;
  const bool local = idx >= 28;
  const bool ab = !comm && !local;
  const int i1 = ab ? (idx & 1) : 0, cl = idx >> 1;
  const int d1 = ab ? cl % 7 : 0;
  const unsigned p1 = ab ? ((unsigned)(rank + 1 + d1) & 7u) : (unsigned)rank;
  const int c1 = ab ? ((g + d1) & 7) * 2 + cl / 7 : g * 2 + (local ? idx - 28 : bid >> 3);
  const int r1 = ab ? i1 * H1 : (comm ? H1B : 0);
  const bool small1 = i1 || local;
  unsigned call = 0;

  if (comm) {
    constexpr int NPC = (int)(XSLOT / 2048), NW = NCOMM * 8, NPA = H1B * 16 * 4 / NW, NPT = NPC / NW;
    static_assert(NPC % NW == 0 && (H1B * 16 * 4) % NW == 0, "x pieces");
    const rsrc_t sr = mk_rsrc(P.x + (size_t)RA * KD * 2, XSLOT);
    const int gw = bid * 8 + wid;
    auto poff = [&](int j) { const int pc = j * NW + gw + HB * 16 * 4; return (pc >= NPC ? pc - NPC : pc) * 2048; };
    u32x4 v[NPT][2];
#pragma unroll
    for (int i = 0; i < 2; ++i) v[0][i] = __builtin_amdgcn_raw_buffer_load_b128(sr, lane * 16 + i * 1024, poff(0), 0);
    typedef __attribute__((address_space(1))) unsigned gu32;
    gu32* xc = (gu32*)(P.ws[(rank + 1 + (lane < 7 ? lane : 0)) & 7] + WS_XCNT + rank * 128);
    if (lane == 0) call = __hip_atomic_fetch_add((unsigned*)(myws + WS_CALL + bid * 64 + wid * 4), 1u, __ATOMIC_RELAXED, __HIP_MEMORY_SCOPE_AGENT) + 1;
    call = RFL(call);
    asm volatile("" : "+v"(xc));
#pragma unroll
    for (int j = 1; j < NPT; ++j)
#pragma unroll
      for (int i = 0; i < 2; ++i) v[j][i] = __builtin_amdgcn_raw_buffer_load_b128(sr, lane * 16 + i * 1024, poff(j), 0);
    const unsigned t0 = (unsigned)NOW();
    if (bid == 0 && tid == 0) __hip_atomic_store((u64*)(myws + WS_TST), ((u64)call << 32) | t0, __ATOMIC_RELAXED, __HIP_MEMORY_SCOPE_AGENT);
#pragma unroll
    for (int j = 0; j < NPT; ++j) {
      if (j == NPT - 1 && tid == 0) {
        const unsigned now = (unsigned)NOW();
        __hip_atomic_store((u64*)(myws + WS_XEND) + bid, ((u64)call << 32) | (now + (now - t0) / (NPT - 1)), __ATOMIC_RELAXED, __HIP_MEMORY_SCOPE_AGENT);
      }
      for (int e = 0; e < 7; ++e) {
        const unsigned pp = (unsigned)(rank + 1 + (e + gw) % 7) & 7u;
        const rsrc_t dr = mk_rsrc(P.ws[pp] + WS_X + ((call & 1) * NR + rank) * XSLOT, XSLOT);
#pragma unroll
        for (int i = 0; i < 2; ++i) __builtin_amdgcn_raw_buffer_store_b128(v[j][i], dr, lane * 16 + i * 1024, poff(j), AUX_SYS);
      }
      if (j == NPA) WAIT_V(14);
      if (j < NPT - 1) BAR();
      if (j == NPA && wid == 1 && lane < 7) __hip_atomic_fetch_add(xc + 16, 1u, __ATOMIC_RELAXED, __HIP_MEMORY_SCOPE_SYSTEM);
    }
    WAIT_V(0);
    BAR();
    if (wid == 7) {
      if (lane < 7) __hip_atomic_fetch_add(xc, 1u, __ATOMIC_RELAXED, __HIP_MEMORY_SCOPE_SYSTEM);
      return;
    }
  }

  if (wid == 7) {
    for (int i0 = (bid - NCOMM) * 64; i0 < MR * KD * 2 / 128; i0 += 240 * 64) {
      const int i = i0 + lane;
      __builtin_amdgcn_global_load_lds((const unsigned*)(P.x + (size_t)(i & 511) * (KD * 2) + (i >> 9) * 128), (LAS unsigned*)(lds + CTL_OFF + 1024), 4, 0, 0);
    }
    return;
  }

  if (wid == 6) {
    constexpr int S1 = NKT, SLAST = 2 * NKT + 1;
    if (comm) {
      if (bid != 0) return;
      for (int s = 0; s <= NKT; ++s) BAR();
      unsigned* of = (unsigned*)(myws + WS_OCNT + (lane & 7) * 128);
      const u64 t1 = NOW();
      for (unsigned it = 1;; ++it) {
        const unsigned v = __hip_atomic_load(of, __ATOMIC_RELAXED, __HIP_MEMORY_SCOPE_SYSTEM);
        if (__all((lane & 7) == rank || v >= call * 32u)) break;
        if ((it & 31u) != 0u) continue;
        if (RFL(__hip_atomic_load(tmo, __ATOMIC_RELAXED, __HIP_MEMORY_SCOPE_AGENT)) == call || NOW() - t1 > DEADLINE) {
          __hip_atomic_store(tmo, call, __ATOMIC_RELAXED, __HIP_MEMORY_SCOPE_AGENT);
          for (int i = 0; i < 20000; ++i) __builtin_amdgcn_s_sleep(127);
          if (lane == 0) *(unsigned short*)(P.out[rank] + (size_t)rank * MR * NS * 2) = 0x7fc0;
          break;
        }
      }
      return;
    }
    constexpr int SLB = SLOT * H1B / H1;
    const int soff = (c1 & 1) * 8 * (SLOT + SLB) + (i1 ? (c1 >> 1) * SLB : 8 * SLB + (c1 >> 1) * SLOT);
    bool tok = !ab, pushed = !ab, sent = !ab;
    unsigned te = 0, d = 0;
    if (lane == 0) { ctl[1] = 0; ctl[2] = 0; }
    WAIT_L0();
    BAR();
    int s = 0;
    {
      unsigned c = 1u; u64 k0 = 0;
      atomics_paced(myws + WS_CALL + bid * 64, c, myws + WS_CALL + bid * 64 + 8, k0, s, S1 - 20);
      call = RFL(c) + 1;
      if (lane == 0) ctl[0] = call;
    }
    while (s < S1 - 20) { BAR(); ++s; }
    const unsigned ta = (unsigned)NOW();
    while (s < S1 - 10 && !tok) {
      const u64 v = poll_paced(myws + WS_TST, s, S1 - 10);
      if (RFL((unsigned)(v >> 32)) == call) { tok = true; te = RFL((unsigned)v); }
      else if (s < S1 - 10) { BAR(); ++s; }
    }
    while (s < S1 - 10) { BAR(); ++s; }
    if (ab && tok) {
      const unsigned tb = (unsigned)NOW();
      const int e = (int)(tb + (tb - ta) * 21 / 20 + PMARG - soff - te);
      d = e < 0 ? 0u : (unsigned)e;
    }
    unsigned tk = 1u;
    u64 key = (ab && tok) ? (((u64)call << 32) | d) : 0ull;
    atomics_paced(myws + WS_TICK + g * 128, tk, myws + WS_BASE + d1 * 128, key, s, S1);
    tk = RFL(tk) % 30u + 2u;
    const Tile2 t2 = tile2(g, (int)tk);
    const unsigned q2 = (unsigned)(rank + 1 + (t2.mt >> 1)) & 7u;
    const char* pw = myws + (lane == 0 ? WS_XCNT + q2 * 128 + (t2.mt & 1) * 64 : lane == 1 ? WS_TST : lane == 2 ? WS_BASE + d1 * 128 : WS_XEND + (lane & 15) * 8);
    bool xok = !t2.peer, xend = false;
    unsigned tlook = (unsigned)NOW() - 100000u;
    auto look = [&](int smax) {
      tlook = (unsigned)NOW();
      const u64 v = poll_paced(pw, s, smax);
      xok = xok || __builtin_amdgcn_readlane((unsigned)v, 0) >= call * (unsigned)NCOMM;
      if (!tok && __builtin_amdgcn_readlane((unsigned)(v >> 32), 1) == call) { tok = true; te = __builtin_amdgcn_readlane((unsigned)v, 1); }
      if (__builtin_amdgcn_readlane((unsigned)(v >> 32), 2) == call) { const unsigned o = __builtin_amdgcn_readlane((unsigned)v, 2); d = o > d ? o : d; }
      if (!xend && tok && (unsigned)(__ballot((unsigned)(v >> 32) == call) >> 16 & 0xffffu) == 0xffffu) {
        xend = true;
        for (int i = 16; i < 32; ++i) { const unsigned o = __builtin_amdgcn_readlane((unsigned)v, i) - te; d = o > d ? o : d; }
      }
    };
    auto stale = [&]() { const unsigned slot = te + d + soff; return (int)((unsigned)NOW() + 150u - slot) >= 0 && (!xend || (int)(tlook + 250u - slot) < 0); };
    auto due = [&]() { return tok && (int)((unsigned)NOW() - (te + d + soff)) >= 0 && !stale() && RFL(ctl[2]) == 4u; };
    auto push = [&]() {
      CFENCE();
      const rsrc_t dr = mk_rsrc(P.out[p1] + ((size_t)(rank * MR + i1 * BM1) * NS + c1 * BN) * 2, ((i1 ? H1B : H1) * 16 - 1) * NS * 2 + 256);
      const int r0 = lane >> 4;
      const int o0 = STG_OFF + r0 * 256 + (((lane & 15) ^ r0) << 4), o1 = STG_OFF + (r0 + 4) * 256 + (((lane & 15) ^ (r0 + 4)) << 4);
      u32x4 v[H1 * 4];
#pragma unroll
      for (int i = 0; i < H1 * 4; ++i) v[i] = *(LAS u32x4*)(lds + ((i & 1) ? o1 : o0) + (i >> 1) * 2048);
#pragma unroll
      for (int i = 0; i < H1B * 4; ++i) __builtin_amdgcn_raw_buffer_store_b128(v[i], dr, r0 * (NS * 2) + (lane & 15) * 16, i * 4 * NS * 2, AUX_SYS);
      if (!i1) {
#pragma unroll
        for (int i = H1B * 4; i < H1 * 4; ++i) __builtin_amdgcn_raw_buffer_store_b128(v[i], dr, r0 * (NS * 2) + (lane & 15) * 16, i * 4 * NS * 2, AUX_SYS);
      }
      pushed = true;
    };
    auto signal = [&]() {
      if (lane == 0) __hip_atomic_fetch_add((unsigned*)(P.ws[p1] + WS_OCNT + rank * 128), 1u, __ATOMIC_RELAXED, __HIP_MEMORY_SCOPE_SYSTEM);
      sent = true;
    };
    const u64 t0 = NOW();
    while (!xok) {
      if (!pushed && s >= S1 && due()) push();
      look(S1);
      if (pushed && !sent) signal();
      if (xok) break;
      if (s < S1) { BAR(); ++s; continue; }
      if (RFL(__hip_atomic_load(tmo, __ATOMIC_RELAXED, __HIP_MEMORY_SCOPE_AGENT)) == call || NOW() - t0 > DEADLINE) {
        __hip_atomic_store(tmo, call, __ATOMIC_RELAXED, __HIP_MEMORY_SCOPE_AGENT);
        break;
      }
    }
    if (t2.peer) { asm volatile("buffer_inv sc0 sc1" ::: "memory"); pace(s, S1); }
    if (lane == 0) ctl[1] = tk + 1;
    WAIT_L0();
    while (!pushed) {
      const bool lk = !tok || stale();
      if (lk) look(SLAST);
      if ((s >= SLAST && NOW() - t0 > DEADLINE) || (s >= S1 && due())) { push(); break; }
      if (s >= SLAST) __builtin_amdgcn_s_sleep(1);
      else if (!lk || !tok) { BAR(); ++s; }
    }
    if (!sent) { pace(s, SLAST); signal(); }
    return;
  }

  if (wid >= 4) {
    const int lw = wid - 4;
    {
      const char* b1 = P.w + (size_t)(p1 * NS + c1 * BN) * 2;
      const char* a1 = P.x + (size_t)r1 * 16 * KD * 2;
      if (small1) load_tile<H1B, H1B, NST1>(lds, a1, P.x, b1, lw, lane);
      else load_tile<H1, H1, NST1>(lds, a1, P.x, b1, lw, lane);
      if (comm) return;
    }
    unsigned tk;
    while ((tk = RFL(ctl[1])) == 0u) __builtin_amdgcn_s_sleep(1);
    call = RFL(ctl[0]);
    const Tile2 t2 = tile2(g, (int)tk - 1);
    const unsigned q2 = (unsigned)(rank + 1 + (t2.mt >> 1)) & 7u;
    const char* b2 = P.w + (size_t)(rank * NS + t2.c2 * BN) * 2;
    const char* own = P.x + (size_t)(t2.own < 0 ? 0 : t2.own) * 16 * KD * 2;
    const char* a = t2.peer ? myws + WS_X + ((call & 1) * NR + q2) * XSLOT + (size_t)(t2.mt & 1) * HB * 16 * KD * 2 : own;
    if (t2.big) load_tile<HB + 2, HB, 3>(lds, a, (t2.peer && t2.own >= 0) ? own : a + (size_t)HB * 16 * KD * 2, b2, lw, lane);
    else load_tile<HB, HB, 4>(lds, a, a, b2, lw, lane);
    return;
  }

  const int wr = wid >> 1, wc = wid & 1, fr = lane & 15, fq = lane >> 4;
  const int coff = (wc * 64 + (fq & 1) * 16 + (fq >> 1) * 8) * 2;
  char* const o = P.out[rank];
  auto round1 = [&](auto mtc) {
    constexpr int MT = decltype(mtc)::value;
    f32x4 acc[MT][4];
    BAR();
    mfma_tile<MT, NST1>(acc, lds, wr, wc, lane);
    if (!ab) {
      char* dst = o + ((size_t)(rank * MR + (r1 + wr * MT) * 16 + fr) * NS + c1 * BN) * 2 + coff;
#pragma unroll
      for (int m = 0; m < MT; ++m)
#pragma unroll
        for (int u = 0; u < 2; ++u) *(u32x4*)(dst + (size_t)m * 16 * NS * 2 + u * 64) = pack8(acc[m][2 * u], acc[m][2 * u + 1]);
    } else {
#pragma unroll
      for (int m = 0; m < MT; ++m)
#pragma unroll
        for (int u = 0; u < 2; ++u)
          *(LAS u32x4*)(lds + STG_OFF + (wr * MT * 16 + m * 16 + fr) * 256 + ((((coff >> 4) + u * 4) ^ (fr & 7)) << 4)) = pack8(acc[m][2 * u], acc[m][2 * u + 1]);
      WAIT_L0();
      if (lane == 0) __hip_atomic_fetch_add((LAS unsigned*)(lds + CTL_OFF + 8), 1u, __ATOMIC_RELAXED, __HIP_MEMORY_SCOPE_WORKGROUP);
    }
  };
  if (small1) round1(std::integral_constant<int, H1B / 2>{});
  else round1(std::integral_constant<int, H1 / 2>{});
  if (comm) return;
  BAR();
  call = RFL(ctl[0]);
  const Tile2 t2 = tile2(g, (int)RFL(ctl[1]) - 1);
  const unsigned tw = __hip_atomic_load(tmo, __ATOMIC_RELAXED, __HIP_MEMORY_SCOPE_AGENT);
  auto round2 = [&](auto mtc, auto nstc) {
    constexpr int MT = decltype(mtc)::value;
    f32x4 acc[MT][4];
    mfma_tile<MT, decltype(nstc)::value>(acc, lds, wr, wc, lane);
    const bool bad = tw == call;
    const int q2 = (rank + 1 + (t2.mt >> 1)) & 7;
#pragma unroll
    for (int m = 0; m < MT; ++m) {
      const int s = wr * MT + m;
      const int row = !t2.peer ? rank * MR + (t2.own + s) * 16
                               : ((s < HB || t2.own < 0) ? q2 * MR + RA + ((t2.mt & 1) * HB + s) * 16 : rank * MR + (t2.own + s - HB) * 16);
      char* dst = o + ((size_t)(row + fr) * NS + t2.c2 * BN) * 2 + coff;
#pragma unroll
      for (int u = 0; u < 2; ++u) {
        u32x4 v = pack8(acc[m][2 * u], acc[m][2 * u + 1]);
        if (bad) v = u32x4{0x7fc07fc0u, 0x7fc07fc0u, 0x7fc07fc0u, 0x7fc07fc0u};
        asm volatile("global_store_dwordx4 %0, %1, off sc1\n\ts_nop 1" ::"v"(dst + u * 64), "v"(v) : "memory");
      }
    }
  };
  if (t2.big) round2(std::integral_constant<int, HB / 2 + 1>{}, std::integral_constant<int, 3>{});
  else round2(std::integral_constant<int, HB / 2>{}, std::integral_constant<int, 4>{});
}
}

extern "C" void kernel_launch(int rank, int nranks, ncclComm_t comm, hipStream_t stream,
                              void* const* d_in, const int* in_sizes, int n_in,
                              void* d_out, int out_size, const int* in_dtypes, int out_dtype,
                              void* const* d_out_all, void* const* d_ws_all, size_t ws_bytes) {
  if (nranks != NR || (unsigned)rank >= (unsigned)NR || n_in < 2 || ws_bytes < WS_NEED ||
      in_sizes[0] != MR * KD || in_sizes[1] != KD * ND || out_size != NR * MR * NS) {
    fprintf(stderr, "kernel_launch: unexpected problem shape\n");
    return;
  }
  Params P;
  P.x = (const char*)d_in[0]; P.w = (const char*)d_in[1]; P.rank = rank;
  for (int p = 0; p < NR; ++p) { P.out[p] = (char*)d_out_all[p]; P.ws[p] = (char*)d_ws_all[p]; }
  gemm_a2a<<<256, 512, 0, stream>>>(P);
}
```

```cpp
#include <hip/hip_runtime.h>
#include <rccl/rccl.h>
#include <cstdio>
#include <type_traits>

namespace {
typedef __bf16 bf16x2 __attribute__((ext_vector_type(2)));
typedef __bf16 bf16x4 __attribute__((ext_vector_type(4)));
typedef __bf16 bf16x8 __attribute__((ext_vector_type(8)));
typedef float f32x2 __attribute__((ext_vector_type(2)));
typedef float f32x4 __attribute__((ext_vector_type(4)));
typedef unsigned u32x4 __attribute__((ext_vector_type(4)));
typedef unsigned long long u64;
#define LAS __attribute__((address_space(3)))

constexpr int NR = 8, MR = 512, KD = 4096, ND = 16384, NS = 2048;
constexpr int BK = 64, NKT = KD / BK, BN = 128;
constexpr int H1 = 10, H1B = 8, HB = 6;
constexpr int BM1 = 16 * H1, RA = 16 * (H1 + H1B), RB = MR - RA;
constexpr int BBYTES = BK * BN * 2;
constexpr int STG_OFF = 4 * (HB * 2048 + BBYTES);
constexpr int CTL_OFF = STG_OFF + H1 * 4096;
constexpr int LDS_BYTES = 160 * 1024;
static_assert(3 * ((HB + 2) * 2048 + BBYTES) <= STG_OFF && 4 * (H1 * 2048 + BBYTES) <= CTL_OFF && CTL_OFF + 2048 <= LDS_BYTES, "lds");
static_assert(RB == (2 * HB + 2) * 16 && H1B == HB + 2, "split");
constexpr int NCOMM = 16;
constexpr int SLOT = 84;
constexpr int PMARG = 110;
constexpr int NST1 = 3;
constexpr int NMOVE = 1;
constexpr int ADT0 = 3, ADB = 6, ADHI = 62, ADLO = 50;

constexpr size_t WS_CALL = 0;
constexpr size_t WS_XCNT = 32768;
constexpr size_t WS_OCNT = 36864;
constexpr size_t WS_TMO = 40960;
constexpr size_t WS_TST = 45056;
constexpr size_t WS_XEND = 45312;
constexpr size_t WS_TICK = 49152;
constexpr size_t WS_BASE = 53248;
constexpr size_t WS_X = 1u << 20;
constexpr size_t XSLOT = (size_t)RB * KD * 2;
constexpr size_t WS_NEED = WS_X + 2 * NR * XSLOT;
constexpr u64 DEADLINE = 80000000ull;

struct Params {
  const char* x; const char* w;
  char* out[NR]; char* ws[NR];
  int rank;
};

#define WAIT_V(n) asm volatile("s_waitcnt vmcnt(%0)" ::"n"(n) : "memory")
#define WAIT_L0() __builtin_amdgcn_s_waitcnt(0xC07F)
#define CFENCE() asm volatile("" ::: "memory")
#define BAR() do { CFENCE(); __builtin_amdgcn_s_barrier(); CFENCE(); } while (0)
#define NOW() __builtin_amdgcn_s_memrealtime()
#define RFL(x) __builtin_amdgcn_readfirstlane(x)

__device__ __forceinline__ unsigned pk(float a, float b) { f32x2 v = {a, b}; return __builtin_bit_cast(unsigned, __builtin_convertvector(v, bf16x2)); }
typedef __amdgpu_buffer_rsrc_t rsrc_t;
__device__ __forceinline__ rsrc_t mk_rsrc(const void* base, unsigned bytes) {
  const u64 b = (u64)base;
  const unsigned lo = RFL((unsigned)b), hi = RFL((unsigned)(b >> 32));
  return __builtin_amdgcn_make_buffer_rsrc((void*)(((u64)hi << 32) | lo), 0, bytes, 0x00020000);
}
constexpr int AUX_SYS = 17;

#define PACE_LOOP                                     \
  "1:\n\t"                                            \
  "s_getreg_b32 %[t], hwreg(HW_REG_IB_STS)\n\t"       \
  "s_and_b32 %[t], %[t], 0xc0000f\n\t"                \
  "s_cbranch_scc0 3f\n\t"                             \
  "s_cmp_ge_i32 %[s], %[m]\n\t"                       \
  "s_cbranch_scc1 2f\n\t"                             \
  "s_barrier\n\t"                                     \
  "s_add_i32 %[s], %[s], 1\n\t"                       \
  "s_branch 1b\n\t"                                   \
  "2:\n\t"                                            \
  "s_sleep 1\n\t"                                     \
  "s_branch 1b\n\t"                                   \
  "3:\n\t"                                            \
  "s_waitcnt vmcnt(0)"
__device__ __forceinline__ void pace(int& s, int smax) {
  unsigned t0; s = RFL(s);
  asm volatile(PACE_LOOP : [s] "+s"(s), [t] "=&s"(t0) : [m] "s"(smax) : "memory", "scc");
}
__device__ __forceinline__ u64 poll_paced(const void* p, int& s, int smax) {
  unsigned t0; u64 v; s = RFL(s);
  asm volatile("global_load_dwordx2 %[v], %[p], off sc0 sc1\n\t" PACE_LOOP
               : [s] "+s"(s), [t] "=&s"(t0), [v] "=&v"(v) : [m] "s"(smax), [p] "v"(p) : "memory", "scc");
  return v;
}
__device__ __forceinline__ void atomics_paced(void* pa, unsigned& a, void* pk_, u64& k, int& s, int smax) {
  unsigned t0; u64 ex; s = RFL(s);
  asm volatile("s_mov_b64 %[ex], exec\n\t"
               "s_mov_b64 exec, 1\n\t"
               "global_atomic_add %[a], %[pa], %[a], off sc0\n\t"
               "global_atomic_umax_x2 %[k], %[pk], %[k], off sc0\n\t"
               "s_mov_b64 exec, %[ex]\n\t" PACE_LOOP
               : [s] "+s"(s), [t] "=&s"(t0), [ex] "=&s"(ex), [a] "+v"(a), [k] "+v"(k) : [m] "s"(smax), [pa] "v"(pa), [pk] "v"(pk_) : "memory", "scc");
}

template <int MT> struct Frag { bf16x8 a[MT]; bf16x8 b[4]; };

template <int MT>
__device__ __forceinline__ void rd(Frag<MT>& f, LAS char* base, const int (&ab)[2], const int (&bb)[4], int ks) {
  f.a[0] = *(LAS bf16x8*)(base + ab[ks]);
#pragma unroll
  for (int n = 0; n < 4; ++n) {
    bf16x4 lo = __builtin_amdgcn_ds_read_tr16_b64_v4bf16((LAS bf16x4*)(base + bb[n] + (ks * 32) * 256));
    bf16x4 hi = __builtin_amdgcn_ds_read_tr16_b64_v4bf16((LAS bf16x4*)(base + bb[n] + (ks * 32 + 4) * 256));
    f.b[n] = __builtin_shufflevector(lo, hi, 0, 1, 2, 3, 4, 5, 6, 7);
  }
#pragma unroll
  for (int m = 1; m < MT; ++m) f.a[m] = *(LAS bf16x8*)(base + ab[ks] + m * 2048);
}
template <int MT, int M0, int M1>
__device__ __forceinline__ void mm(f32x4 (&acc)[MT][4], const Frag<MT>& f) {
#pragma unroll
  for (int m = M0; m < M1; ++m)
#pragma unroll
    for (int j = 0; j < 4; ++j) { const int n = (m & 1) ? 3 - j : j; acc[m][n] = __builtin_amdgcn_mfma_f32_16x16x32_bf16(f.b[n], f.a[m], acc[m][n], 0, 0, 0); }
}
template <int NMF, int NRD>
__device__ __forceinline__ void sched() {
  constexpr int NI = NRD < NMF ? NRD : NMF, DBL = NRD - NI;
#pragma unroll
  for (int i = 0; i < NI; ++i) {
    __builtin_amdgcn_sched_group_barrier(0x8, 1, 0); __builtin_amdgcn_sched_group_barrier(0x100, 1, 0);
    if (i < DBL) __builtin_amdgcn_sched_group_barrier(0x100, 1, 0);
  }
  if constexpr (NMF > NI) __builtin_amdgcn_sched_group_barrier(0x8, NMF - NI, 0);
}
__device__ __forceinline__ u32x4 pack8(const f32x4& c0, const f32x4& c1) {
  unsigned x0 = pk(c0[0], c0[1]), x1 = pk(c0[2], c0[3]), y0 = pk(c1[0], c1[1]), y1 = pk(c1[2], c1[3]);
  auto s0 = __builtin_amdgcn_permlane16_swap(x0, y0, false, false);
  auto s1 = __builtin_amdgcn_permlane16_swap(x1, y1, false, false);
  return u32x4{s0[0], s1[0], s0[1], s1[1]};
}

template <int MT, int NST>
__device__ __forceinline__ void mfma_tile(f32x4 (&acc)[MT][4], LAS char* lds, int wr, int wc, int lane) {
  constexpr int H = 2 * MT, STB = H * 2048 + BBYTES;
  const int fr = lane & 15, fq = lane >> 4;
  int ab[2], bb[4];
#pragma unroll
  for (int ks = 0; ks < 2; ++ks) ab[ks] = (wr * MT * 16 + fr) * 128 + (((ks * 4 + fq) ^ ((fr >> 1) & 7)) << 4);
  {
    const int q = (lane & 15) >> 2, p = lane & 3, k = 8 * fq + q;
    const int sw = (k & 3) | (((k >> 3) & 1) << 2);
#pragma unroll
    for (int n = 0; n < 4; ++n) bb[n] = H * 2048 + k * 256 + (((wc * 4 + n) ^ sw) << 5) + p * 8;
  }
#pragma unroll
  for (int m = 0; m < MT; ++m)
#pragma unroll
    for (int n = 0; n < 4; ++n) acc[m][n] = f32x4{0.f, 0.f, 0.f, 0.f};
  Frag<MT> f0, f1;
  rd<MT>(f0, lds, ab, bb, 0);
  WAIT_L0();
  int so = 0;
  for (int t = 0; t < NKT - 1; ++t) {
    rd<MT>(f1, lds + so, ab, bb, 1);
    mm<MT, 0, MT>(acc, f0);
    sched<MT * 4, MT + 8>();
    __builtin_amdgcn_sched_barrier(0);
    mm<MT, 0, 1>(acc, f1);
    __builtin_amdgcn_sched_barrier(0);
    WAIT_L0();
    BAR();
    so = (so + STB == NST * STB) ? 0 : so + STB;
    rd<MT>(f0, lds + so, ab, bb, 0);
    mm<MT, 1, MT>(acc, f1);
    sched<MT * 4 - 4, MT + 8>();
  }
  rd<MT>(f1, lds + so, ab, bb, 1);
  mm<MT, 0, MT>(acc, f0);
  sched<MT * 4, MT + 8>();
  __builtin_amdgcn_sched_barrier(0);
  WAIT_L0();
  BAR();
  mm<MT, 0, MT>(acc, f1);
}

template <int H, int HA, int NST>
__device__ __forceinline__ void load_tile(LAS char* lds, const char* a, const char* a2, const char* b, int lw, int lane, bool adapt = false) {
  constexpr int STB = H * 2048 + BBYTES, D = H + 8;
  const int arow = lw * 8 + (lane >> 3);
  const unsigned va = (unsigned)(arow * (KD * 2) + (((lane & 7) ^ ((arow >> 1) & 7)) << 4));
  unsigned vb[2];
#pragma unroll
  for (int i = 0; i < 2; ++i) {
    const int k = (lw + 2 * i) * 4 + (lane >> 4);
    const int sw = (k & 3) | (((k >> 3) & 1) << 2);
    vb[i] = (unsigned)(k * (ND * 2) + (((lane & 15) ^ (sw << 1)) << 4));
  }
  int so = 0;
  const rsrc_t ra = mk_rsrc(a, HA * 16 * KD * 2), ra2 = mk_rsrc(a2, (H - HA) * 16 * KD * 2), rb = mk_rsrc(b, (unsigned)((KD - 1) * ND * 2 + 256));
  auto issue_aux = [&](int t, auto auxc) {
    LAS char* dst = lds + so + lw * 1024;
    so = (so + STB == NST * STB) ? 0 : so + STB;
#pragma unroll
    for (int i = 0; i < 8; ++i)
      __builtin_amdgcn_raw_ptr_buffer_load_lds(rb, dst + H * 2048 + i * 2048, 16, vb[i & 1], (t * BK + (i >> 1) * 16) * (ND * 2), 0, decltype(auxc)::value);
#pragma unroll
    for (int i = 0; i < H; ++i)
      __builtin_amdgcn_raw_ptr_buffer_load_lds(i < HA ? ra : ra2, dst + i * 2048, 16, va, t * 128 + (i < HA ? i : i - HA) * 16 * KD * 2, 0, 0);
  };
  auto issue = [&](int t) { issue_aux(t, std::integral_constant<int, 0>{}); };
  issue(0); issue(1);
  WAIT_V(D); BAR();
#pragma unroll
  for (int t = 2; t < NST; ++t) issue(t);
  int t = 0;
  if (adapt) {
    bool nt = false;
    unsigned dref = 0;
#pragma nounroll
    for (; t < ADT0; ++t) { WAIT_V((NST - 2) * D); BAR(); issue(t + NST); }
    while (t + ADB <= NKT - NST) {
      const unsigned ta = (unsigned)NOW();
      const int te = t + ADB;
      if (nt) {
#pragma nounroll
        for (; t < te; ++t) { WAIT_V((NST - 2) * D); BAR(); issue_aux(t + NST, std::integral_constant<int, 2>{}); }
      } else {
#pragma nounroll
        for (; t < te; ++t) { WAIT_V((NST - 2) * D); BAR(); issue(t + NST); }
      }
      const unsigned dt = (unsigned)NOW() - ta;
      if (!nt) { if (dt > (unsigned)(ADHI * ADB) && dref == 0u) { nt = true; dref = dt; } }
      else if (dt < (unsigned)(ADLO * ADB) || dt > dref) nt = false;
    }
  }
  for (; t < NKT - NST; ++t) { WAIT_V((NST - 2) * D); BAR(); issue(t + NST); }
  if constexpr (NST == 4) { WAIT_V(2 * D); BAR(); }
  WAIT_V(D); BAR();
  WAIT_V(0); BAR();
  BAR();
}

struct Tile2 { int big, peer, c2, mt, own; };
__device__ __forceinline__ Tile2 tile2(int g, int tk) {
  Tile2 t;
  const int par = g & 1, j = tk >> 2;
  t.c2 = (g >> 1) * 4 + ((tk & 3) ^ (par ? (j == 6 ? 3 : 2) : 0));
  t.big = tk < (par ? 24 + NMOVE : 24 - NMOVE); t.peer = j > 0;
  const unsigned tab = par ? 0xCA8DB970u : 0x64205310u;
  t.mt = (tab >> (4 * j)) & 15;
  t.own = j == 0 ? 18 : (!t.big ? -1 : (j == 5 ? 26 + par * 4 : (j == 6 ? 26 : ((j == 4 && !par) ? 28 : -1))));
  return t;
}

__global__ __launch_bounds__(512, 2) void gemm_a2a(Params P) {
  __shared__ __attribute__((aligned(1024))) char smem[LDS_BYTES];
  LAS char* lds = (LAS char*)smem;
  const int tid = threadIdx.x, lane = tid & 63;
  const int wid = RFL(tid >> 6);
  const int rank = P.rank;
  char* const myws = P.ws[rank];
  unsigned* const tmo = (unsigned*)(myws + WS_TMO);
  volatile LAS unsigned* ctl = (volatile LAS unsigned*)(lds + CTL_OFF);
  const int bid = blockIdx.x;
  const bool comm = bid < NCOMM;
  const int g = bid & 7, idx = (bid >> 3) - 2;
  const bool local = idx >= 28;
  const bool ab = !comm && !local;
  const int i1 = ab ? (idx & 1) : 0, cl = idx >> 1;
  const int d1 = ab ? cl % 7 : 0;
  const unsigned p1 = ab ? ((unsigned)(rank + 1 + d1) & 7u) : (unsigned)rank;
  const int c1 = ab ? ((g + d1) & 7) * 2 + cl / 7 : g * 2 + (local ? idx - 28 : bid >> 3);
  const int r1 = ab ? i1 * H1 : (comm ? H1B : 0);
  const bool small1 = i1 || local;
  unsigned call = 0;

  if (comm) {
    constexpr int NPC = (int)(XSLOT / 2048), NW = NCOMM * 8, NPA = H1B * 16 * 4 / NW, NPT = NPC / NW;
    static_assert(NPC % NW == 0 && (H1B * 16 * 4) % NW == 0, "x pieces");
    const rsrc_t sr = mk_rsrc(P.x + (size_t)RA * KD * 2, XSLOT);
    const int gw = bid * 8 + wid;
    auto poff = [&](int j) { const int pc = j * NW + gw + HB * 16 * 4; return (pc >= NPC ? pc - NPC : pc) * 2048; };
    u32x4 v[NPT][2];
#pragma unroll
    for (int i = 0; i < 2; ++i) v[0][i] = __builtin_amdgcn_raw_buffer_load_b128(sr, lane * 16 + i * 1024, poff(0), 0);
    typedef __attribute__((address_space(1))) unsigned gu32;
    gu32* xc = (gu32*)(P.ws[(rank + 1 + (lane < 7 ? lane : 0)) & 7] + WS_XCNT + rank * 128);
    if (lane == 0) call = __hip_atomic_fetch_add((unsigned*)(myws + WS_CALL + bid * 64 + wid * 4), 1u, __ATOMIC_RELAXED, __HIP_MEMORY_SCOPE_AGENT) + 1;
    call = RFL(call);
    asm volatile("" : "+v"(xc));
#pragma unroll
    for (int j = 1; j < NPT; ++j)
#pragma unroll
      for (int i = 0; i < 2; ++i) v[j][i] = __builtin_amdgcn_raw_buffer_load_b128(sr, lane * 16 + i * 1024, poff(j), 0);
    const unsigned t0 = (unsigned)NOW();
    if (bid == 0 && tid == 0) __hip_atomic_store((u64*)(myws + WS_TST), ((u64)call << 32) | t0, __ATOMIC_RELAXED, __HIP_MEMORY_SCOPE_AGENT);
#pragma unroll
    for (int j = 0; j < NPT; ++j) {
      if (j == NPT - 1 && tid == 0) {
        const unsigned now = (unsigned)NOW();
        __hip_atomic_store((u64*)(myws + WS_XEND) + bid, ((u64)call << 32) | (now + (now - t0) / (NPT - 1)), __ATOMIC_RELAXED, __HIP_MEMORY_SCOPE_AGENT);
      }
      for (int e = 0; e < 7; ++e) {
        const unsigned pp = (unsigned)(rank + 1 + (e + gw) % 7) & 7u;
        const rsrc_t dr = mk_rsrc(P.ws[pp] + WS_X + ((call & 1) * NR + rank) * XSLOT, XSLOT);
#pragma unroll
        for (int i = 0; i < 2; ++i) __builtin_amdgcn_raw_buffer_store_b128(v[j][i], dr, lane * 16 + i * 1024, poff(j), AUX_SYS);
      }
      if (j == NPA) WAIT_V(14);
      if (j < NPT - 1) BAR();
      if (j == NPA && wid == 1 && lane < 7) __hip_atomic_fetch_add(xc + 16, 1u, __ATOMIC_RELAXED, __HIP_MEMORY_SCOPE_SYSTEM);
    }
    WAIT_V(0);
    BAR();
    if (wid == 7) {
      if (lane < 7) __hip_atomic_fetch_add(xc, 1u, __ATOMIC_RELAXED, __HIP_MEMORY_SCOPE_SYSTEM);
      return;
    }
  }

  if (wid == 7) {
    for (int i0 = (bid - NCOMM) * 64; i0 < MR * KD * 2 / 128; i0 += 240 * 64) {
      const int i = i0 + lane;
      __builtin_amdgcn_global_load_lds((const unsigned*)(P.x + (size_t)(i & 511) * (KD * 2) + (i >> 9) * 128), (LAS unsigned*)(lds + CTL_OFF + 1024), 4, 0, 0);
    }
    return;
  }

  if (wid == 6) {
    constexpr int S1 = NKT, SLAST = 2 * NKT + 1;
    if (comm) {
      if (bid != 0) return;
      for (int s = 0; s <= NKT; ++s) BAR();
      unsigned* of = (unsigned*)(myws + WS_OCNT + (lane & 7) * 128);
      const u64 t1 = NOW();
      for (unsigned it = 1;; ++it) {
        const unsigned v = __hip_atomic_load(of, __ATOMIC_RELAXED, __HIP_MEMORY_SCOPE_SYSTEM);
        if (__all((lane & 7) == rank || v >= call * 32u)) break;
        if ((it & 31u) != 0u) continue;
        if (RFL(__hip_atomic_load(tmo, __ATOMIC_RELAXED, __HIP_MEMORY_SCOPE_AGENT)) == call || NOW() - t1 > DEADLINE) {
          __hip_atomic_store(tmo, call, __ATOMIC_RELAXED, __HIP_MEMORY_SCOPE_AGENT);
          for (int i = 0; i < 20000; ++i) __builtin_amdgcn_s_sleep(127);
          if (lane == 0) *(unsigned short*)(P.out[rank] + (size_t)rank * MR * NS * 2) = 0x7fc0;
          break;
        }
      }
      return;
    }
    constexpr int SLB = SLOT * H1B / H1;
    const int soff = (c1 & 1) * 8 * (SLOT + SLB) + (i1 ? (c1 >> 1) * SLB : 8 * SLB + (c1 >> 1) * SLOT);
    bool tok = !ab, pushed = !ab, sent = !ab;
    unsigned te = 0, d = 0;
    if (lane == 0) { ctl[1] = 0; ctl[2] = 0; }
    WAIT_L0();
    BAR();
    int s = 0;
    {
      unsigned c = 1u; u64 k0 = 0;
      atomics_paced(myws + WS_CALL + bid * 64, c, myws + WS_CALL + bid * 64 + 8, k0, s, S1 - 20);
      call = RFL(c) + 1;
      if (lane == 0) ctl[0] = call;
    }
    while (s < S1 - 20) { BAR(); ++s; }
    const unsigned ta = (unsigned)NOW();
    while (s < S1 - 10 && !tok) {
      const u64 v = poll_paced(myws + WS_TST, s, S1 - 10);
      if (RFL((unsigned)(v >> 32)) == call) { tok = true; te = RFL((unsigned)v); }
      else if (s < S1 - 10) { BAR(); ++s; }
    }
    while (s < S1 - 10) { BAR(); ++s; }
    if (ab && tok) {
      const unsigned tb = (unsigned)NOW();
      const int e = (int)(tb + (tb - ta) * 21 / 20 + PMARG - soff - te);
      d = e < 0 ? 0u : (unsigned)e;
    }
    unsigned tk = 1u;
    u64 key = (ab && tok) ? (((u64)call << 32) | d) : 0ull;
    atomics_paced(myws + WS_TICK + g * 128, tk, myws + WS_BASE + d1 * 128, key, s, S1);
    tk = RFL(tk) % 30u + 2u;
    const Tile2 t2 = tile2(g, (int)tk);
    const unsigned q2 = (unsigned)(rank + 1 + (t2.mt >> 1)) & 7u;
    const char* pw = myws + (lane == 0 ? WS_XCNT + q2 * 128 + (t2.mt & 1) * 64 : lane == 1 ? WS_TST : lane == 2 ? WS_BASE + d1 * 128 : WS_XEND + (lane & 15) * 8);
    bool xok = !t2.peer, xend = false;
    unsigned tlook = (unsigned)NOW() - 100000u;
    auto look = [&](int smax) {
      tlook = (unsigned)NOW();
      const u64 v = poll_paced(pw, s, smax);
      xok = xok || __builtin_amdgcn_readlane((unsigned)v, 0) >= call * (unsigned)NCOMM;
      if (!tok && __builtin_amdgcn_readlane((unsigned)(v >> 32), 1) == call) { tok = true; te = __builtin_amdgcn_readlane((unsigned)v, 1); }
      if (__builtin_amdgcn_readlane((unsigned)(v >> 32), 2) == call) { const unsigned o = __builtin_amdgcn_readlane((unsigned)v, 2); d = o > d ? o : d; }
      if (!xend && tok && (unsigned)(__ballot((unsigned)(v >> 32) == call) >> 16 & 0xffffu) == 0xffffu) {
        xend = true;
        for (int i = 16; i < 32; ++i) { const unsigned o = __builtin_amdgcn_readlane((unsigned)v, i) - te; d = o > d ? o : d; }
      }
    };
    auto stale = [&]() { const unsigned slot = te + d + soff; return (int)((unsigned)NOW() + 150u - slot) >= 0 && (!xend || (int)(tlook + 250u - slot) < 0); };
    auto due = [&]() { return tok && (int)((unsigned)NOW() - (te + d + soff)) >= 0 && !stale() && RFL(ctl[2]) == 4u; };
    auto push = [&]() {
      CFENCE();
      const rsrc_t dr = mk_rsrc(P.out[p1] + ((size_t)(rank * MR + i1 * BM1) * NS + c1 * BN) * 2, ((i1 ? H1B : H1) * 16 - 1) * NS * 2 + 256);
      const int r0 = lane >> 4;
      const int o0 = STG_OFF + r0 * 256 + (((lane & 15) ^ r0) << 4), o1 = STG_OFF + (r0 + 4) * 256 + (((lane & 15) ^ (r0 + 4)) << 4);
      u32x4 v[H1 * 4];
#pragma unroll
      for (int i = 0; i < H1 * 4; ++i) v[i] = *(LAS u32x4*)(lds + ((i & 1) ? o1 : o0) + (i >> 1) * 2048);
#pragma unroll
      for (int i = 0; i < H1B * 4; ++i) __builtin_amdgcn_raw_buffer_store_b128(v[i], dr, r0 * (NS * 2) + (lane & 15) * 16, i * 4 * NS * 2, AUX_SYS);
      if (!i1) {
#pragma unroll
        for (int i = H1B * 4; i < H1 * 4; ++i) __builtin_amdgcn_raw_buffer_store_b128(v[i], dr, r0 * (NS * 2) + (lane & 15) * 16, i * 4 * NS * 2, AUX_SYS);
      }
      pushed = true;
    };
    auto signal = [&]() {
      if (lane == 0) __hip_atomic_fetch_add((unsigned*)(P.ws[p1] + WS_OCNT + rank * 128), 1u, __ATOMIC_RELAXED, __HIP_MEMORY_SCOPE_SYSTEM);
      sent = true;
    };
    const u64 t0 = NOW();
    while (!xok) {
      if (!pushed && s >= S1 && due()) push();
      look(S1);
      if (pushed && !sent) signal();
      if (xok) break;
      if (s < S1) { BAR(); ++s; continue; }
      if (RFL(__hip_atomic_load(tmo, __ATOMIC_RELAXED, __HIP_MEMORY_SCOPE_AGENT)) == call || NOW() - t0 > DEADLINE) {
        __hip_atomic_store(tmo, call, __ATOMIC_RELAXED, __HIP_MEMORY_SCOPE_AGENT);
        break;
      }
    }
    if (t2.peer) { asm volatile("buffer_inv sc0 sc1" ::: "memory"); pace(s, S1); }
    if (lane == 0) ctl[1] = tk + 1;
    WAIT_L0();
    while (!pushed) {
      const bool lk = !tok || stale();
      if (lk) look(SLAST);
      if ((s >= SLAST && NOW() - t0 > DEADLINE) || (s >= S1 && due())) { push(); break; }
      if (s >= SLAST) __builtin_amdgcn_s_sleep(1);
      else if (!lk || !tok) { BAR(); ++s; }
    }
    if (!sent) { pace(s, SLAST); signal(); }
    return;
  }

  if (wid >= 4) {
    const int lw = wid - 4;
    {
      const char* b1 = P.w + (size_t)(p1 * NS + c1 * BN) * 2;
      const char* a1 = P.x + (size_t)r1 * 16 * KD * 2;
      if (small1) load_tile<H1B, H1B, NST1>(lds, a1, P.x, b1, lw, lane, ab);
      else load_tile<H1, H1, NST1>(lds, a1, P.x, b1, lw, lane, ab);
      if (comm) return;
    }
    unsigned tk;
    while ((tk = RFL(ctl[1])) == 0u) __builtin_amdgcn_s_sleep(1);
    call = RFL(ctl[0]);
    const Tile2 t2 = tile2(g, (int)tk - 1);
    const unsigned q2 = (unsigned)(rank + 1 + (t2.mt >> 1)) & 7u;
    const char* b2 = P.w + (size_t)(rank * NS + t2.c2 * BN) * 2;
    const char* own = P.x + (size_t)(t2.own < 0 ? 0 : t2.own) * 16 * KD * 2;
    const char* a = t2.peer ? myws + WS_X + ((call & 1) * NR + q2) * XSLOT + (size_t)(t2.mt & 1) * HB * 16 * KD * 2 : own;
    if (t2.big) load_tile<HB + 2, HB, 3>(lds, a, (t2.peer && t2.own >= 0) ? own : a + (size_t)HB * 16 * KD * 2, b2, lw, lane);
    else load_tile<HB, HB, 4>(lds, a, a, b2, lw, lane);
    return;
  }

  const int wr = wid >> 1, wc = wid & 1, fr = lane & 15, fq = lane >> 4;
  const int coff = (wc * 64 + (fq & 1) * 16 + (fq >> 1) * 8) * 2;
  char* const o = P.out[rank];
  auto round1 = [&](auto mtc) {
    constexpr int MT = decltype(mtc)::value;
    f32x4 acc[MT][4];
    BAR();
    mfma_tile<MT, NST1>(acc, lds, wr, wc, lane);
    if (!ab) {
      char* dst = o + ((size_t)(rank * MR + (r1 + wr * MT) * 16 + fr) * NS + c1 * BN) * 2 + coff;
#pragma unroll
      for (int m = 0; m < MT; ++m)
#pragma unroll
        for (int u = 0; u < 2; ++u) *(u32x4*)(dst + (size_t)m * 16 * NS * 2 + u * 64) = pack8(acc[m][2 * u], acc[m][2 * u + 1]);
    } else {
#pragma unroll
      for (int m = 0; m < MT; ++m)
#pragma unroll
        for (int u = 0; u < 2; ++u)
          *(LAS u32x4*)(lds + STG_OFF + (wr * MT * 16 + m * 16 + fr) * 256 + ((((coff >> 4) + u * 4) ^ (fr & 7)) << 4)) = pack8(acc[m][2 * u], acc[m][2 * u + 1]);
      WAIT_L0();
      if (lane == 0) __hip_atomic_fetch_add((LAS unsigned*)(lds + CTL_OFF + 8), 1u, __ATOMIC_RELAXED, __HIP_MEMORY_SCOPE_WORKGROUP);
    }
  };
  if (small1) round1(std::integral_constant<int, H1B / 2>{});
  else round1(std::integral_constant<int, H1 / 2>{});
  if (comm) return;
  BAR();
  call = RFL(ctl[0]);
  const Tile2 t2 = tile2(g, (int)RFL(ctl[1]) - 1);
  const unsigned tw = __hip_atomic_load(tmo, __ATOMIC_RELAXED, __HIP_MEMORY_SCOPE_AGENT);
  auto round2 = [&](auto mtc, auto nstc) {
    constexpr int MT = decltype(mtc)::value;
    f32x4 acc[MT][4];
    mfma_tile<MT, decltype(nstc)::value>(acc, lds, wr, wc, lane);
    const bool bad = tw == call;
    const int q2 = (rank + 1 + (t2.mt >> 1)) & 7;
#pragma unroll
    for (int m = 0; m < MT; ++m) {
      const int s = wr * MT + m;
      const int row = !t2.peer ? rank * MR + (t2.own + s) * 16
                               : ((s < HB || t2.own < 0) ? q2 * MR + RA + ((t2.mt & 1) * HB + s) * 16 : rank * MR + (t2.own + s - HB) * 16);
      char* dst = o + ((size_t)(row + fr) * NS + t2.c2 * BN) * 2 + coff;
#pragma unroll
      for (int u = 0; u < 2; ++u) {
        u32x4 v = pack8(acc[m][2 * u], acc[m][2 * u + 1]);
        if (bad) v = u32x4{0x7fc07fc0u, 0x7fc07fc0u, 0x7fc07fc0u, 0x7fc07fc0u};
        asm volatile("global_store_dwordx4 %0, %1, off sc1\n\ts_nop 1" ::"v"(dst + u * 64), "v"(v) : "memory");
      }
    }
  };
  if (t2.big) round2(std::integral_constant<int, HB / 2 + 1>{}, std::integral_constant<int, 3>{});
  else round2(std::integral_constant<int, HB / 2>{}, std::integral_constant<int, 4>{});
}
}

extern "C" void kernel_launch(int rank, int nranks, ncclComm_t comm, hipStream_t stream,
                              void* const* d_in, const int* in_sizes, int n_in,
                              void* d_out, int out_size, const int* in_dtypes, int out_dtype,
                              void* const* d_out_all, void* const* d_ws_all, size_t ws_bytes) {
  if (nranks != NR || (unsigned)rank >= (unsigned)NR || n_in < 2 || ws_bytes < WS_NEED ||
      in_sizes[0] != MR * KD || in_sizes[1] != KD * ND || out_size != NR * MR * NS) {
    fprintf(stderr, "kernel_launch: unexpected problem shape\n");
    return;
  }
  Params P;
  P.x = (const char*)d_in[0]; P.w = (const char*)d_in[1]; P.rank = rank;
  for (int p = 0; p < NR; ++p) { P.out[p] = (char*)d_out_all[p]; P.ws[p] = (char*)d_ws_all[p]; }
  gemm_a2a<<<256, 512, 0, stream>>>(P);
}
```
